```python
import numpy as np

kernel_name = "gpu_ag_gemm_m2048_k2048_n2048_bf16_none"


ABSMAX_RTOL = 0.008

COMPUTE_DTYPE = "bfloat16"
EPILOGUE = "none"
IN_QUANT = None
OUT_QUANT = None
IN_DTYPES = {"x": "bfloat16", "w_mat": "bfloat16"}
WORLD_SIZE = 8

def _to_bf16(x):
    u = np.ascontiguousarray(x, dtype=np.float32).view(np.uint32)
    return ((u + 0x7FFF + ((u >> 16) & 1)) & 0xFFFF0000).view(np.float32)

def setup_inputs():

    x = _to_bf16(np.random.randn(2048, 2048).astype(np.float32))
    w_mat = _to_bf16((np.random.randn(2048, 2048) / np.sqrt(2048)).astype(np.float32))
    return {"x": x, "w_mat": w_mat}

def _epilogue(y):
    return y

def reference(x, w_mat):
    y = (x @ w_mat).astype(np.float32)
    return _epilogue(y)
```

```cpp
#include <hip/hip_runtime.h>
#include <rccl/rccl.h>
#include <cstdint>
#include <cstdio>

#define HIP_CHECK(expr) do { hipError_t _e = (expr); if (_e != hipSuccess) { \
    fprintf(stderr, "HIP error %s at %s:%d\n", hipGetErrorString(_e), __FILE__, __LINE__); \
    return; } } while (0)

typedef unsigned int u32;
typedef unsigned long long u64;
typedef u32 u32x2 __attribute__((ext_vector_type(2)));
typedef u32 u32x4 __attribute__((ext_vector_type(4)));
typedef float f32x4 __attribute__((ext_vector_type(4)));
typedef float f32x16 __attribute__((ext_vector_type(16)));
typedef __bf16 bf16x8 __attribute__((ext_vector_type(8)));
typedef __bf16 bf16x2 __attribute__((ext_vector_type(2)));
typedef __attribute__((address_space(1))) u32 gu32;
typedef __attribute__((address_space(3))) u32 lu32;

constexpr int NR = 8;
constexpr size_t KiB = 1024, MiB = 1024 * 1024;
constexpr size_t PS = 6 * MiB;
constexpr size_t XS_OFF = 0;
constexpr size_t WS_OFF = 2 * MiB;
constexpr size_t YS_OFF = 3 * MiB;
constexpr size_t YE_OFF = 5 * MiB;
constexpr size_t CTRL = 14 * MiB;
constexpr size_t XF_OFF = CTRL;
constexpr size_t TF_OFF = CTRL + 64 * KiB;
constexpr size_t WF_OFF = CTRL + 96 * KiB;
constexpr size_t CNT_OFF = CTRL + 128 * KiB;
constexpr size_t TMO_OFF = CTRL + 160 * KiB;
constexpr size_t DBG_OFF = CTRL + 256 * KiB;
constexpr size_t WS_NEED = CTRL + 512 * KiB;
constexpr u64 DEADLINE = 50000000ull;
constexpr int L_B = 0, L_RED = 128 * 1024, L_CTL = 144 * 1024, L_BYTES = 144 * 1024 + 512;

struct Params {
  const uint16_t* x;
  const uint16_t* w;
  uint16_t* out;
  char* my;
  char* ws[NR];
  int rank;
};

#define RLX_SYS __ATOMIC_RELAXED, __HIP_MEMORY_SCOPE_SYSTEM
#define RLX_AGT __ATOMIC_RELAXED, __HIP_MEMORY_SCOPE_AGENT
#define BARRIER() asm volatile("s_waitcnt lgkmcnt(0)\n\ts_barrier" ::: "memory")

__device__ __forceinline__ u64 now() { return __builtin_amdgcn_s_memrealtime(); }
__device__ __forceinline__ void st16_wt(void* p, u32x4 v) {
  asm volatile("global_store_dwordx4 %0, %1, off sc0 sc1\n\ts_nop 1" :: "v"(p), "v"(v) : "memory");
}
__device__ __forceinline__ void st_flag(char* p, u32 v) { __hip_atomic_store((gu32*)p, v, RLX_SYS); }
__device__ __forceinline__ u32 ld_flag(const char* p) { return __hip_atomic_load((gu32*)p, RLX_SYS); }
__device__ __forceinline__ u32 lds_ld(int off) {
  u32 v; asm volatile("ds_read_b32 %0, %1\n\ts_waitcnt lgkmcnt(0)" : "=v"(v) : "v"(off) : "memory");
  return __builtin_amdgcn_readfirstlane(v);
}
__device__ __forceinline__ void lds_st(int off, u32 v) { asm volatile("ds_write_b32 %0, %1" :: "v"(off), "v"(v) : "memory"); }
__device__ __forceinline__ const char* uni(const char* p) {
  u64 a = (u64)p;
  u32 lo = __builtin_amdgcn_readfirstlane((u32)a), hi = __builtin_amdgcn_readfirstlane((u32)(a >> 32));
  return (const char*)(((u64)hi << 32) | lo);
}
__device__ __forceinline__ void load4(u32x4* A, u32 voff, const char* base) {
  asm volatile("s_nop 4\n\t"
               "global_load_dwordx4 %0, %4, %5\n\t"
               "global_load_dwordx4 %1, %4, %5 offset:1024\n\t"
               "global_load_dwordx4 %2, %4, %5 offset:2048\n\t"
               "global_load_dwordx4 %3, %4, %5 offset:3072"
               : "=&v"(A[0]), "=&v"(A[1]), "=&v"(A[2]), "=&v"(A[3])
               : "v"(voff), "s"(base) : "memory");
}
__device__ __forceinline__ void dma4(u32 voff, const char* base, u32 lds) {
  u32 keep;
  asm volatile("s_mov_b32 %0, m0\n\ts_mov_b32 m0, %3\n\ts_nop 4\n\t"
               "global_load_lds_dwordx4 %1, %2\n\t"
               "global_load_lds_dwordx4 %1, %2 offset:1024\n\t"
               "global_load_lds_dwordx4 %1, %2 offset:2048\n\t"
               "global_load_lds_dwordx4 %1, %2 offset:3072\n\t"
               "s_mov_b32 m0, %0"
               : "=&s"(keep) : "v"(voff), "s"(base), "s"(lds) : "memory");
}
__device__ __forceinline__ void dma1(u32 voff, const char* base, u32 lds) {
  u32 keep;
  asm volatile("s_mov_b32 %0, m0\n\ts_mov_b32 m0, %3\n\ts_nop 4\n\t"
               "global_load_lds_dwordx4 %1, %2\n\t"
               "s_mov_b32 m0, %0"
               : "=&s"(keep) : "v"(voff), "s"(base), "s"(lds) : "memory");
}
template <int N> __device__ __forceinline__ void vmwait() { asm volatile("s_waitcnt vmcnt(%0)" :: "i"(N) : "memory"); }
template <int N> __device__ __forceinline__ u32 get_call(u32 cnt) {
  u32 c; asm volatile("s_waitcnt vmcnt(%2)\n\tv_readfirstlane_b32 %0, %1\n\ts_nop 1" : "=s"(c) : "v"(cnt), "i"(N) : "memory");
  return c + 1;
}
__device__ __forceinline__ u32 pack2(float lo, float hi) {
  bf16x2 v; v[0] = (__bf16)lo; v[1] = (__bf16)hi;
  return __builtin_bit_cast(u32, v);
}
__device__ __forceinline__ float bflo(u32 v) { return __builtin_bit_cast(float, v << 16); }
__device__ __forceinline__ float bfhi(u32 v) { return __builtin_bit_cast(float, v & 0xffff0000u); }

#define STAMP(k) do {} while (0)
#define STAMP_FLUSH() do {} while (0)

#define WIN 8
#define PSLEEP 10
#define LOOKA 2
#define TAU0 4
#define TAU1 7
#define PFD1 80
#define WD2 40
#define WD3 140
constexpr int push_nflags(int t) { return (t & 3) != 3 ? 0 : 1 + ((t >> 2) == TAU0) + ((t >> 2) == TAU1); }
constexpr int push_younger(int t) {
  int n = 0;
  for (int u = t + 1; u < 32 && u < t + WIN; ++u) ++n;
  for (int u = t - WIN + 1; u < t; ++u) if (u >= 0) n += push_nflags(u);
  return n;
}
template <int T>
__device__ __forceinline__ void push_steps(u32x4* v, char* dx, char* fx, char* tf, u32 call, int lane) {
  if constexpr (T < 32 + WIN) {
    if constexpr (T >= WIN) {
      constexpr int t = T - WIN;
      if constexpr (T < 32 || (t & 3) == 3) vmwait<push_younger(t)>();
      if constexpr ((t & 3) == 3) {
        if (lane == 0) {
          st_flag(fx + (t >> 2) * 1024, call);
          if constexpr ((t >> 2) == TAU0) st_flag(tf, call);
          if constexpr ((t >> 2) == TAU1) st_flag(tf + 1024, call);
        }
        if (t == 3) STAMP(16); if (t == 4 * TAU0 + 3) STAMP(17);
      }
    } else {
      vmwait<31>();
    }
    if constexpr (T < 32) { asm volatile("" : "+v"(v[T])); st16_wt(dx + T * KiB, v[T]); if (T == 0) STAMP(1); }
    push_steps<T + 1>(v, dx, fx, tf, call, lane);
  }
}
__device__ __forceinline__ bool expired(char* my, u64 t0, u32& it, u32 call) {
  if (((++it & 255) == 0 && ld_flag(my + TMO_OFF) == call) || now() - t0 > DEADLINE) {
    __hip_atomic_store((gu32*)(my + TMO_OFF), call, RLX_AGT);
    return true;
  }
  if (PSLEEP) __builtin_amdgcn_s_sleep(PSLEEP);
  return false;
}

__global__ __launch_bounds__(512, 2) void ag_gemm_kernel(Params p) {
  extern __shared__ __attribute__((aligned(16))) char smem[];
  const int b = blockIdx.x, tid = threadIdx.x, lane = tid & 63;
  const int wave = __builtin_amdgcn_readfirstlane(tid >> 6);
  const int r = p.rank & 7;
  const int xc = b & 7, role = (b >> 3) & 3, u8 = (b >> 5) & 7;
  char* const my = p.my;
  const u32 lds0 = (u32)(size_t)(lu32*)smem;
  const int C_FAIL = lds0 + L_CTL, C_BD = C_FAIL + 16;
  const u32 voff = lane * 16;
  char* const cntp = my + CNT_OFF + b * 128;
  u32 cnt;
  asm volatile("global_load_dword %0, %1, off sc1" : "=v"(cnt) : "v"(cntp) : "memory");
  if (tid < 32) lds_st(C_FAIL + tid * 4, 0);
  u32 call;

  if (role == 3) {
    if ((u8 & 1) == 0 && wave < 2) {
      const int sub = u8 + wave, h = (xc + sub) & 7;
      char* wsh = p.ws[h];
      const char* src = uni((const char*)p.x + (size_t)((h & 1) * 128 + (sub & 3) * 32) * 4096 + ((h >> 2) * 1024 + (sub >> 2) * 512) * 2);
      const u32 soff = (lane & 31) * 4096 + (lane >> 5) * 16;
      u32x4 v[32];
#pragma unroll
      for (int q = 0; q < 8; ++q)
        asm volatile("s_nop 4\n\t"
                     "global_load_dwordx4 %0, %4, %5 offset:%6\n\t"
                     "global_load_dwordx4 %1, %4, %5 offset:%7\n\t"
                     "global_load_dwordx4 %2, %4, %5 offset:%8\n\t"
                     "global_load_dwordx4 %3, %4, %5 offset:%9"
                     : "=&v"(v[q * 4]), "=&v"(v[q * 4 + 1]), "=&v"(v[q * 4 + 2]), "=&v"(v[q * 4 + 3])
                     : "v"(soff), "s"(src), "i"(q * 128), "i"(q * 128 + 32), "i"(q * 128 + 64), "i"(q * 128 + 96) : "memory");
      call = get_call<32>(cnt);
      push_steps<0>(v, wsh + (call & 1) * PS + XS_OFF + (size_t)r * 256 * KiB + sub * 32 * KiB + lane * 16,
                    wsh + XF_OFF + (r * 64 + sub) * 128, wsh + TF_OFF + (r * 16 + sub) * 128, call, lane);
      if (wave == 0) STAMP(2);
      vmwait<0>();
    } else {
      call = get_call<0>(cnt);
    }
    BARRIER();
    if (tid == 0) __hip_atomic_store((gu32*)cntp, call, RLX_AGT);
  } else if (role < 2) {
    const int si = role * 64 + u8 * 8 + xc, piece = si >> 5, kk = si & 31;
    const int kstep = 2 * kk + (wave >> 2), g2 = wave & 3;
    const uint16_t* src = p.w + (size_t)((piece >> 1) * 1024 + kstep * 16 + (lane >> 5) * 8) * 256 + (piece & 1) * 128 + g2 * 32 + (lane & 31);
    char* wd0 = p.ws[2 * (piece & 1) + 4 * ((piece >> 1) & 1)];
    char* wd1 = p.ws[2 * (piece & 1) + 4 * ((piece >> 1) & 1) + 1];
    u32 e[8];
#pragma unroll
    for (int t = 0; t < 8; ++t) e[t] = src[t * 256];
    call = get_call<8>(cnt);
    u32x4 v = {e[0] | (e[1] << 16), e[2] | (e[3] << 16), e[4] | (e[5] << 16), e[6] | (e[7] << 16)};
    asm volatile("" : "+s"(wd0), "+s"(wd1));
    st16_wt(my + (call & 1) * PS + WS_OFF + (size_t)((piece * 4 + g2) * 64 + kstep) * KiB + lane * 16, v);
    vmwait<0>();
    BARRIER();
    if (wave == 0 && lane < 2) st_flag((lane ? wd1 : wd0) + WF_OFF + r * 128 + kk * 4, call);
    __builtin_amdgcn_s_waitcnt(0x0F70);
  } else {
    BARRIER();
    call = get_call<0>(cnt);
  }
  const size_t slot = (call & 1) * PS;
  const u32 tag = (call % 65535u + 1u) << 16;
  const u64 t0 = now();
  u32 it = 0;

  const int j = xc;
  const char* const wrem = p.ws[j] + slot + WS_OFF + (size_t)((r >> 2) * 2 + ((r >> 1) & 1)) * 256 * KiB;
#define WAIT_WF() do { while (__ballot(ld_flag(my + WF_OFF + j * 128 + (lane & 31) * 4) >= call) != ~0ull) if (expired(my, t0, it, call)) { lds_st(C_FAIL, 1); break; } } while (0)

  if (role == 3) {
    if (u8 & 1) {
      while (ld_flag(my + TF_OFF + (j * 16 + u8) * 128) < call) if (expired(my, t0, it, call)) break;
      const u64 tp = now() + (wave >> 2) * PFD1;
      WAIT_WF();
      while (now() < tp) __builtin_amdgcn_s_sleep(1);
      if (wave == 0) STAMP(3);
      u32x4 pf[4];
      load4(pf, voff, uni(wrem + (wave >> 2) * 64 * KiB + ((u8 >> 1) * 4 + (wave & 3)) * 4 * KiB));
      vmwait<0>();
      asm volatile("" :: "v"(pf[0]), "v"(pf[1]), "v"(pf[2]), "v"(pf[3]));
      if (wave == 0) STAMP(6);
      if (wave == 4) STAMP(9);
    }
    const int u = xc * 8 + u8, ua = u & 1, ub = (u >> 1) & 1, par = (u >> 2) & 1, i2 = u >> 3;
    const int m2 = wave >> 1, kh2 = wave & 1;
    const bool tl = (lane & 3) == 3;
#pragma unroll 1
    for (int tile = 0; tile < 2; ++tile) {
      const int gam = par + 2 * tile;
      const int unit0 = (((ua + 2 * ub) * 4 + gam) * 8 + i2) * 8 + wave, unit1 = unit0 + 4 * 256;
      const char* e0 = my + slot + YE_OFF + unit0 * 64 + (lane >> 2) * 4;
      const char* e1 = e0 + 4 * 256 * 64;
      const char* y0p = my + slot + YS_OFF + (size_t)unit0 * KiB + lane * 16;
      const char* y1p = my + slot + YS_OFF + (size_t)unit1 * KiB + lane * 16;
      u32x4 o = {0x7fc07fc0u, 0x7fc07fc0u, 0x7fc07fc0u, 0x7fc07fc0u};
      while (true) {
        u32 x0 = tag, x1 = tag;
        if (tl) { x0 = ld_flag(e0); x1 = ld_flag(e1); }
        if (__ballot(((x0 ^ tag) | (x1 ^ tag)) >> 16 == 0) == ~0ull) {
          if (wave == 2) { if (tile == 0) STAMP(19); else STAMP(14); }
          u32x4 y0, y1; u32 tm;
          asm volatile("global_load_dwordx4 %0, %3, off sc0 sc1\n\tglobal_load_dwordx4 %1, %4, off sc0 sc1\n\t"
                       "global_load_dword %2, %5, off sc1\n\ts_waitcnt vmcnt(0)"
                       : "=&v"(y0), "=&v"(y1), "=&v"(tm) : "v"(y0p), "v"(y1p), "v"(my + TMO_OFF) : "memory");
          if (__ballot(tm == call)) break;
          if (__ballot(!tl || ((y0[3] ^ tag) | (y1[3] ^ tag)) >> 16 == 0) == ~0ull) {
            if (tl) { y0[3] = (y0[3] & 0xffffu) | (x0 << 16); y1[3] = (y1[3] & 0xffffu) | (x1 << 16); }
#pragma unroll
            for (int t = 0; t < 4; ++t) o[t] = pack2(bflo(y0[t]) + bflo(y1[t]), bfhi(y0[t]) + bfhi(y1[t]));
            break;
          }
        }
        if (expired(my, t0, it, call)) break;
      }
      *(u32x4*)(p.out + (size_t)(i2 * 256 + ua * 128 + m2 * 32 + (lane & 31)) * 256 + ub * 128 + gam * 32 + kh2 * 16 + (lane >> 5) * 8) = o;
    }
    STAMP(15);
    STAMP_FLUSH();
    return;
  }

  const int i = u8;
  const int m = wave & 3, kh = wave >> 2;
  const char* abase = uni(my + slot + XS_OFF + (size_t)i * 256 * KiB + wave * 32 * KiB);
  const char* fq = my + XF_OFF + (i * 64 + (lane & 7) * 8 + wave) * 128;
  char* const own = p.ws[j] + slot;
  u32x4 A[32];
  u32 rdy = 0;
#define XWAIT(q) do { while (rdy <= (u32)(q)) { \
      u32 v_ = 0; if ((u32)lane - rdy < (u32)LOOKA && lane < 8) v_ = ld_flag(fq); \
      rdy += __builtin_ctz(~(u32)(__ballot(v_ >= call) >> rdy)); \
      if (rdy <= (u32)(q) && expired(my, t0, it, call)) { lds_st(C_FAIL, 1); rdy = 8; } } } while (0)
#pragma unroll
  for (int q = 0; q < 8; ++q) {
    if (q == 5) WAIT_WF();
    XWAIT(q);
    if (wave == 0) { if (q == 0) STAMP(4); if (q == 3) STAMP(5); if (q == 7) STAMP(7); }
    load4(A + q * 4, voff, abase + q * 4 * KiB);
  }
  u32x4 yv = {}; u32 yex = 0;
#define SEND_Y(g) do { const int unit_ = ((r * 4 + (g)) * 8 + i) * 8 + m * 2 + kh; \
      st16_wt(own + YS_OFF + (size_t)unit_ * KiB + lane * 16, yv); \
      if ((lane & 3) == 3) asm volatile("global_store_dword %0, %1, off sc0 sc1" :: "v"(own + YE_OFF + unit_ * 64 + (lane >> 2) * 4), "v"(yex) : "memory"); } while (0)
  const int ntile = role == 0 ? 2 : 1;
  int gam = role == 0 ? 0 : role + 1;
  if (role != 0) {
    while (ld_flag(my + TF_OFF + (j * 16 + 8 + i) * 128) < call) if (expired(my, t0, it, call)) { lds_st(C_FAIL, 1); break; }
    const u64 tp = now() + (role == 1 ? WD2 : WD3);
    while (now() < tp) __builtin_amdgcn_s_sleep(1);
  }
  if (wave == 0) STAMP(8);
  {
    const char* wb = uni(wrem + (size_t)(gam * 64 + wave * 8) * KiB);
    const u32 lb = lds0 + L_B + wave * 8192;
    dma4(voff, wb, lb); dma4(voff, wb + 4 * KiB, lb + 4096);
    if (role == 0) { dma4(voff, wb + 64 * KiB, lb + 65536); dma4(voff, wb + 68 * KiB, lb + 65536 + 4096); }
  }
#define PUBLISH(t, c) do { if (lane == 0) asm volatile("ds_add_u32 %0, %1" :: "v"(C_BD + (t) * 4), "v"(1 << ((c) * 8 + wave)) : "memory"); } while (0)
  if (role == 0) vmwait<12>(); else vmwait<4>();
  PUBLISH(0, 0);
  if (wave == 0) STAMP(24);
#pragma unroll 1
  for (int tile = 0; tile < ntile; ++tile, ++gam) {
    const int C_T = C_BD + tile * 4;
#pragma unroll
    for (int q = 0; q < 8; ++q)
      asm volatile("" : "+v"(A[q * 4 + 0]), "+v"(A[q * 4 + 1]), "+v"(A[q * 4 + 2]), "+v"(A[q * 4 + 3]) :: "memory");
    f32x16 acc = {};
    const char* bl = smem + L_B + tile * 65536 + kh * 32 * 1024 + lane * 16;
    u32 st = 0;
#define BWAIT(mask) do { while ((st & ((mask) << (kh * 4))) != ((mask) << (kh * 4))) st = lds_ld(C_T); } while (0)
    bf16x8 wf[2][8];
#pragma unroll
    for (int c = 0; c < 2; ++c) {
      if (c == 1 && tile == 0) {
        if (role == 0) vmwait<8>(); else vmwait<0>();
        PUBLISH(0, 1);
        if (wave == 0) STAMP(26);
      }
      BWAIT(3u << (c * 8));
#pragma unroll
      for (int s = 0; s < 8; ++s) wf[0][s] = *(const bf16x8*)(bl + ((s >> 2) * 8 + c * 4 + (s & 3)) * 1024);
      BWAIT(12u << (c * 8));
#pragma unroll
      for (int s = 0; s < 8; ++s) wf[1][s] = *(const bf16x8*)(bl + (16 + (s >> 2) * 8 + c * 4 + (s & 3)) * 1024);
      __builtin_amdgcn_sched_barrier(0);
#pragma unroll
      for (int s = 0; s < 16; ++s)
        acc = __builtin_amdgcn_mfma_f32_32x32x16_bf16(wf[s >> 3][s & 7], __builtin_bit_cast(bf16x8, A[(s >> 2) * 8 + c * 4 + (s & 3)]), acc, 0, 0, 0);
      __builtin_amdgcn_sched_barrier(0);
    }
    if (wave == 0) { if (tile == 0) STAMP(10); else STAMP(12); }
    if (wave == 7) { if (tile == 1) STAMP(20); else STAMP(21); }
    const bool more = tile + 1 < ntile;
    if (more) { vmwait<0>(); PUBLISH(1, 0); PUBLISH(1, 1); if (wave == 0) STAMP(25); }
    f32x4 o0, o1, k0, k1;
    if (kh == 0) { k0 = acc.s0123; k1 = acc.s4567; o0 = acc.s89ab; o1 = acc.scdef; }
    else         { o0 = acc.s0123; o1 = acc.s4567; k0 = acc.s89ab; k1 = acc.scdef; }
    char* red = smem + (tile == 0 ? L_RED : L_B);
    *(f32x4*)(red + wave * 2048 + lane * 16) = o0;
    *(f32x4*)(red + wave * 2048 + 1024 + lane * 16) = o1;
    BARRIER();
    if (wave == 0) { if (tile == 0) STAMP(22); else STAMP(23); }
    const u32 failed = *(const u32*)(smem + L_CTL);
    k0 += *(const f32x4*)(red + (wave ^ 4) * 2048 + lane * 16);
    k1 += *(const f32x4*)(red + (wave ^ 4) * 2048 + 1024 + lane * 16);
    u32 d0 = pack2(k0[0], k0[1]), d1 = pack2(k0[2], k0[3]), d2 = pack2(k1[0], k1[1]), d3 = pack2(k1[2], k1[3]);
    auto s0 = __builtin_amdgcn_permlane32_swap(d0, d2, false, false);
    auto s1 = __builtin_amdgcn_permlane32_swap(d1, d3, false, false);
    yv = u32x4{s0[0], s1[0], s0[1], s1[1]};
    if (failed) yv = u32x4{0x7fc07fc0u, 0x7fc07fc0u, 0x7fc07fc0u, 0x7fc07fc0u};
    yex = (yv[3] >> 16) | tag;
    if ((lane & 3) == 3) yv[3] = (yv[3] & 0xffffu) | tag;
    if (more) { SEND_Y(gam); if (wave == 0) STAMP(11); }
  }
  SEND_Y(gam - 1);
  if (wave == 0) STAMP(13);
  if (tid == 0) __hip_atomic_store((gu32*)cntp, call, RLX_AGT);
  STAMP_FLUSH();
}

extern "C" void kernel_launch(int rank, int nranks, ncclComm_t comm, hipStream_t stream,
                              void* const* d_in, const int* in_sizes, int n_in,
                              void* d_out, int out_size, const int* in_dtypes, int out_dtype,
                              void* const* d_out_all, void* const* d_ws_all, size_t ws_bytes) {
  if (nranks != NR || n_in < 2 || ws_bytes < WS_NEED || (unsigned)rank >= (unsigned)NR ||
      in_sizes[0] != 256 * 2048 || in_sizes[1] != 2048 * 256 || out_size != 2048 * 256) {
    fprintf(stderr, "kernel_launch: unexpected problem shape\n");
    return;
  }
  static bool attr_done = false;
  if (!attr_done) {
    HIP_CHECK(hipFuncSetAttribute((const void*)ag_gemm_kernel, hipFuncAttributeMaxDynamicSharedMemorySize, L_BYTES));
    attr_done = true;
  }
  Params p;
  p.x = (const uint16_t*)d_in[0];
  p.w = (const uint16_t*)d_in[1];
  p.out = (uint16_t*)d_out;
  p.my = (char*)d_ws_all[rank];
  for (int q = 0; q < NR; ++q) p.ws[q] = (char*)d_ws_all[q];
  p.rank = rank;
  ag_gemm_kernel<<<256, 512, L_BYTES, stream>>>(p);
  HIP_CHECK(hipGetLastError());
}
```

```cpp
#include <hip/hip_runtime.h>
#include <rccl/rccl.h>
#include <cstdint>
#include <cstdio>

#define HIP_CHECK(expr) do { hipError_t _e = (expr); if (_e != hipSuccess) { \
    fprintf(stderr, "HIP error %s at %s:%d\n", hipGetErrorString(_e), __FILE__, __LINE__); \
    return; } } while (0)

typedef unsigned int u32;
typedef unsigned long long u64;
typedef u32 u32x2 __attribute__((ext_vector_type(2)));
typedef u32 u32x4 __attribute__((ext_vector_type(4)));
typedef float f32x4 __attribute__((ext_vector_type(4)));
typedef float f32x16 __attribute__((ext_vector_type(16)));
typedef __bf16 bf16x8 __attribute__((ext_vector_type(8)));
typedef __bf16 bf16x2 __attribute__((ext_vector_type(2)));
typedef __attribute__((address_space(1))) u32 gu32;
typedef __attribute__((address_space(3))) u32 lu32;

constexpr int NR = 8;
constexpr size_t KiB = 1024, MiB = 1024 * 1024;
constexpr size_t PS = 7 * MiB;
constexpr size_t XS_OFF = 0;
constexpr size_t WS_OFF = 2 * MiB;
constexpr size_t YS_OFF = 4 * MiB;
constexpr size_t YE_OFF = 6 * MiB;
constexpr size_t CTRL = 2 * PS;
constexpr size_t XF_OFF = CTRL;
constexpr size_t WF_OFF = CTRL + 64 * KiB;
constexpr size_t CNT_OFF = CTRL + 128 * KiB;
constexpr size_t TMO_OFF = CTRL + 160 * KiB;
constexpr size_t DBG_OFF = CTRL + 256 * KiB;
constexpr size_t WS_NEED = CTRL + 512 * KiB;
constexpr u64 DEADLINE = 50000000ull;
constexpr int L_B = 0, L_RED = 128 * 1024, L_CTL = 144 * 1024, L_EX = L_CTL + 512, L_BYTES = L_EX + 1024;

struct Params {
  const uint16_t* x;
  const uint16_t* w;
  uint16_t* out;
  char* my;
  char* ws[NR];
  int rank;
};

#define RLX_SYS __ATOMIC_RELAXED, __HIP_MEMORY_SCOPE_SYSTEM
#define RLX_AGT __ATOMIC_RELAXED, __HIP_MEMORY_SCOPE_AGENT
#define BARRIER() asm volatile("s_waitcnt lgkmcnt(0)\n\ts_barrier" ::: "memory")

__device__ __forceinline__ u64 now() { return __builtin_amdgcn_s_memrealtime(); }
__device__ __forceinline__ void st16_wt(void* p, u32x4 v) {
  asm volatile("global_store_dwordx4 %0, %1, off sc0 sc1\n\ts_nop 1" :: "v"(p), "v"(v) : "memory");
}
__device__ __forceinline__ void st_flag(char* p, u32 v) { __hip_atomic_store((gu32*)p, v, RLX_SYS); }
__device__ __forceinline__ u32 ld_flag(const char* p) { return __hip_atomic_load((gu32*)p, RLX_SYS); }
__device__ __forceinline__ u32 lds_ld(int off) {
  u32 v; asm volatile("ds_read_b32 %0, %1\n\ts_waitcnt lgkmcnt(0)" : "=v"(v) : "v"(off) : "memory");
  return __builtin_amdgcn_readfirstlane(v);
}
__device__ __forceinline__ void lds_st(int off, u32 v) { asm volatile("ds_write_b32 %0, %1" :: "v"(off), "v"(v) : "memory"); }
__device__ __forceinline__ const char* uni(const char* p) {
  u64 a = (u64)p;
  u32 lo = __builtin_amdgcn_readfirstlane((u32)a), hi = __builtin_amdgcn_readfirstlane((u32)(a >> 32));
  return (const char*)(((u64)hi << 32) | lo);
}
#define LSC " sc0 sc1"
__device__ __forceinline__ void load4(u32x4* A, u32 voff, const char* base) {
  asm volatile("s_nop 4\n\t"
               "global_load_dwordx4 %0, %4, %5\n\t"
               "global_load_dwordx4 %1, %4, %5 offset:1024\n\t"
               "global_load_dwordx4 %2, %4, %5 offset:2048\n\t"
               "global_load_dwordx4 %3, %4, %5 offset:3072"
               : "=&v"(A[0]), "=&v"(A[1]), "=&v"(A[2]), "=&v"(A[3])
               : "v"(voff), "s"(base) : "memory");
}
__device__ __forceinline__ void load4v(u32x4* A, u32 v0, u32 v1, u32 v2, u32 v3, const char* base) {
  asm volatile("s_nop 4\n\t"
               "global_load_dwordx4 %0, %4, %8" LSC "\n\t"
               "global_load_dwordx4 %1, %5, %8" LSC "\n\t"
               "global_load_dwordx4 %2, %6, %8" LSC "\n\t"
               "global_load_dwordx4 %3, %7, %8" LSC
               : "=&v"(A[0]), "=&v"(A[1]), "=&v"(A[2]), "=&v"(A[3])
               : "v"(v0), "v"(v1), "v"(v2), "v"(v3), "s"(base) : "memory");
}
__device__ __forceinline__ void dma4(u32 voff, const char* base, u32 lds) {
  u32 keep;
  asm volatile("s_mov_b32 %0, m0\n\ts_mov_b32 m0, %3\n\ts_nop 4\n\t"
               "global_load_lds_dwordx4 %1, %2" LSC "\n\t"
               "global_load_lds_dwordx4 %1, %2 offset:1024" LSC "\n\t"
               "global_load_lds_dwordx4 %1, %2 offset:2048" LSC "\n\t"
               "global_load_lds_dwordx4 %1, %2 offset:3072" LSC "\n\t"
               "s_mov_b32 m0, %0"
               : "=&s"(keep) : "v"(voff), "s"(base), "s"(lds) : "memory");
}
__device__ __forceinline__ void dma1(u32 voff, const char* base, u32 lds) {
  u32 keep;
  asm volatile("s_mov_b32 %0, m0\n\ts_mov_b32 m0, %3\n\ts_nop 4\n\t"
               "global_load_lds_dwordx4 %1, %2\n\t"
               "s_mov_b32 m0, %0"
               : "=&s"(keep) : "v"(voff), "s"(base), "s"(lds) : "memory");
}
template <int N> __device__ __forceinline__ void vmwait() { asm volatile("s_waitcnt vmcnt(%0)" :: "i"(N) : "memory"); }
template <int N> __device__ __forceinline__ u32 get_call(u32 cnt) {
  u32 c; asm volatile("s_waitcnt vmcnt(%2)\n\tv_readfirstlane_b32 %0, %1\n\ts_nop 1" : "=s"(c) : "v"(cnt), "i"(N) : "memory");
  return c + 1;
}
__device__ __forceinline__ u32 pack2(float lo, float hi) {
  bf16x2 v; v[0] = (__bf16)lo; v[1] = (__bf16)hi;
  return __builtin_bit_cast(u32, v);
}
__device__ __forceinline__ float bflo(u32 v) { return __builtin_bit_cast(float, v << 16); }
__device__ __forceinline__ float bfhi(u32 v) { return __builtin_bit_cast(float, v & 0xffff0000u); }

#define STAMP(k) do {} while (0)
#define STAMP_FLUSH() do {} while (0)

#define WIN 6
#define PSLEEP 10
#define WSLEEP 2
#define G0 1040
#define GSTEP 146
#define WDMA_AT 100
#define LOOKA 2
constexpr bool push_flagged(int t) { return (t & 3) == 3; }
constexpr int push_younger(int t) {
  int n = 0;
  for (int u = t + 1; u < 64 && u < t + WIN; ++u) ++n;
  for (int u = t - WIN + 1; u < t; ++u) if (u >= 0 && push_flagged(u)) ++n;
  return n;
}
template <int T>
__device__ __forceinline__ void push_steps(u32x4* v, char* dx, char* dw, char* fx, char* fw, u32 ta, u32 call, int lane) {
  if constexpr (T < 64 + WIN) {
    if constexpr (T >= WIN) {
      constexpr int t = T - WIN;
      if constexpr (T < 64 || push_flagged(t)) vmwait<push_younger(t)>();
      if constexpr (push_flagged(t)) {
        if (lane == 0) st_flag(t < 32 ? fx + (t >> 2) * 1024 : fw + ((t - 32) >> 2) * 1024, call);
        if (t == 3) STAMP(16); if (t == 31) STAMP(17); if (t == 39) STAMP(18);
      }
    } else {
      vmwait<31>();
    }
    if constexpr (T < 32) { asm volatile("" : "+v"(v[T])); st16_wt(dx + T * KiB, v[T]); if (T == 0) STAMP(1); }
    else if constexpr (T < 64) { st16_wt(dw + (((T - 32) >> 3) * 64 + (T & 7)) * KiB, v[T - 32]); }
    if constexpr (T == 16) BARRIER();
    if constexpr (T >= 16 && T < 32) {
      constexpr int f = 2 * (T - 16), off = (f & 7) * 4096 + (f >> 3) * 64;
      u32x2 a0, a1, a2, a3;
      asm volatile("ds_read_b64_tr_b16 %0, %4 offset:%5\n\t"
                   "ds_read_b64_tr_b16 %1, %4 offset:%5+1024\n\t"
                   "ds_read_b64_tr_b16 %2, %4 offset:%5+4096\n\t"
                   "ds_read_b64_tr_b16 %3, %4 offset:%5+5120\n\t"
                   "s_waitcnt lgkmcnt(0)"
                   : "=&v"(a0), "=&v"(a1), "=&v"(a2), "=&v"(a3) : "v"(ta), "i"(off) : "memory");
      v[f] = u32x4{a0[0], a0[1], a1[0], a1[1]};
      v[f + 1] = u32x4{a2[0], a2[1], a3[0], a3[1]};
    }
    push_steps<T + 1>(v, dx, dw, fx, fw, ta, call, lane);
  }
}
template <int SL = PSLEEP>
__device__ __forceinline__ bool expired(char* my, u64 t0, u32& it, u32 call) {
  if ((++it & 63) == 0 && (ld_flag(my + TMO_OFF) == call || now() - t0 > DEADLINE)) {
    __hip_atomic_store((gu32*)(my + TMO_OFF), call, RLX_AGT);
    return true;
  }
  if (SL) __builtin_amdgcn_s_sleep(SL);
  return false;
}

__global__ __launch_bounds__(512, 2) void ag_gemm_kernel(Params p) {
  extern __shared__ __attribute__((aligned(16))) char smem[];
  const int b = blockIdx.x, tid = threadIdx.x, lane = tid & 63;
  const int wave = __builtin_amdgcn_readfirstlane(tid >> 6);
  const int r = p.rank & 7;
  const int xc = b & 7, u8 = (b >> 3) & 7, role = (3 + (b >> 6)) & 3;
  char* const my = p.my;
  char* const peer = p.ws[xc];
  const char* xin = (const char*)p.x;
  asm volatile("" : "+s"(xin));
  const u32 lds0 = (u32)(size_t)(lu32*)smem;
  const int C_FAIL = lds0 + L_CTL, C_BD = C_FAIL + 16;
  const u32 voff = lane * 16;
  const u64 t0 = now();
  char* const cntp = my + CNT_OFF + b * 128;
  u32 cnt;
  asm volatile("global_load_dword %0, %1, off sc1" : "=v"(cnt) : "v"(cntp) : "memory");
  u32 call;

  if (__builtin_expect(role == 3 && wave == 0, 1)) {
    const int h = xc, sub = u8;
    const char* src = uni(xin + (size_t)((h & 1) * 128 + (sub & 3) * 32) * 4096 + ((h >> 2) * 1024 + (sub >> 2) * 512) * 2);
    const u32 soff = (lane & 31) * 4096 + (lane >> 5) * 16;
    u32x4 v[32];
#pragma unroll
    for (int q = 0; q < 8; ++q)
      asm volatile("s_nop 4\n\t"
                   "global_load_dwordx4 %0, %4, %5 offset:%6\n\t"
                   "global_load_dwordx4 %1, %4, %5 offset:%7\n\t"
                   "global_load_dwordx4 %2, %4, %5 offset:%8\n\t"
                   "global_load_dwordx4 %3, %4, %5 offset:%9"
                   : "=&v"(v[q * 4]), "=&v"(v[q * 4 + 1]), "=&v"(v[q * 4 + 2]), "=&v"(v[q * 4 + 3])
                   : "v"(soff), "s"(src), "i"(q * 128), "i"(q * 128 + 32), "i"(q * 128 + 64), "i"(q * 128 + 96) : "memory");
    call = get_call<32>(cnt);
    char* base = peer + (call & 1) * PS + (size_t)r * 256 * KiB + lane * 16;
    push_steps<0>(v, base + XS_OFF + sub * 32 * KiB, base + WS_OFF + sub * 8 * KiB, peer + XF_OFF + (r * 64 + sub) * 128, peer + WF_OFF + (r * 64 + sub) * 128,
                  lds0 + ((lane >> 5) * 8 + ((lane & 15) >> 2)) * 256 + ((lane >> 4) & 1) * 32 + (lane & 3) * 8, call, lane);
    if (lane == 0) __hip_atomic_store((gu32*)cntp, call, RLX_AGT);
    STAMP(2);
    vmwait<0>();
    BARRIER();
  } else {
    if (wave == 1 && lane < 32) lds_st(C_FAIL + lane * 4, 0);
    if (role == 3) {
      if (wave == 1) {
        const char* src = uni((const char*)p.w + (size_t)((xc >> 2) * 1024 + u8 * 128) * 512 + ((xc >> 1) & 1) * 256);
        const u32 soff = (lane >> 4) * 512 + (lane & 15) * 16;
        while (now() < t0 + WDMA_AT) __builtin_amdgcn_s_sleep(4);
#pragma unroll
        for (int t = 0; t < 32; ++t) dma1(soff, src + t * 2048, lds0 + t * 1024);
      }
      call = get_call<0>(cnt);
      BARRIER();
      BARRIER();
    } else {
      BARRIER();
      call = get_call<0>(cnt);
    }
  }
  const size_t slot = (call & 1) * PS;
  const u32 tag = (call % 65535u + 1u) << 16;
  u32 it = 0;

  if (role == 3) {
    const int u = xc * 8 + u8, ua = u & 1, ub = (u >> 1) & 1, par = (u >> 2) & 1, i2 = u >> 3;
    const int m2 = wave >> 1, kh2 = wave & 1;
    const bool tl = (lane & 3) == 3;
#pragma unroll 1
    for (int tile = 0; tile < 2; ++tile) {
      const int gam = par + 2 * tile;
      const int unit0 = (((ua + 2 * ub) * 4 + gam) * 8 + i2) * 8 + wave, unit1 = unit0 + 4 * 256;
      const char* e0 = my + slot + YE_OFF + unit0 * 64 + (lane >> 2) * 4;
      const char* e1 = e0 + 4 * 256 * 64;
      const char* y0p = my + slot + YS_OFF + (size_t)unit0 * KiB + lane * 16;
      const char* y1p = my + slot + YS_OFF + (size_t)unit1 * KiB + lane * 16;
      u32x4 o = {0x7fc07fc0u, 0x7fc07fc0u, 0x7fc07fc0u, 0x7fc07fc0u};
      while (__ballot(!tl || (ld_flag(e0) ^ tag) >> 16 == 0) != ~0ull && __ballot(!tl || (ld_flag(e1) ^ tag) >> 16 == 0) != ~0ull)
        if (expired<WSLEEP>(my, t0, it, call)) break;
      while (true) {
        u32x4 y0, y1; u32 x0, x1, tm;
        asm volatile("global_load_dwordx4 %0, %5, off sc0 sc1\n\tglobal_load_dwordx4 %1, %6, off sc0 sc1\n\t"
                     "global_load_dword %2, %7, off sc0 sc1\n\tglobal_load_dword %3, %8, off sc0 sc1\n\t"
                     "global_load_dword %4, %9, off sc1\n\ts_waitcnt vmcnt(0)"
                     : "=&v"(y0), "=&v"(y1), "=&v"(x0), "=&v"(x1), "=&v"(tm) : "v"(y0p), "v"(y1p), "v"(e0), "v"(e1), "v"(my + TMO_OFF) : "memory");
        if (__ballot(tm == call)) break;
        if (__ballot(!tl || ((y0[3] ^ tag) | (y1[3] ^ tag) | (x0 ^ tag) | (x1 ^ tag)) >> 16 == 0) == ~0ull) {
          if (wave == 2) { if (tile == 0) STAMP(19); else STAMP(14); }
          if (tl) { y0[3] = (y0[3] & 0xffffu) | (x0 << 16); y1[3] = (y1[3] & 0xffffu) | (x1 << 16); }
#pragma unroll
          for (int t = 0; t < 4; ++t) o[t] = pack2(bflo(y0[t]) + bflo(y1[t]), bfhi(y0[t]) + bfhi(y1[t]));
          break;
        }
        if (expired<WSLEEP>(my, t0, it, call)) break;
      }
      uint16_t* op = p.out + (size_t)(i2 * 256 + ua * 128 + m2 * 32 + (lane & 31)) * 256 + ub * 128 + gam * 32 + kh2 * 16 + (lane >> 5) * 8;
      *(u32x4*)op = o;
    }
    STAMP(15);
    STAMP_FLUSH();
    return;
  }

  const int i = u8, j = xc;
  const int m = wave & 3, kh = wave >> 2;
  const bool diag = i == j;
  const char* abase = uni(diag ? xin + (size_t)((j & 1) * 128 + m * 32) * 4096 + ((j >> 2) * 1024 + kh * 512) * 2
                               : my + slot + XS_OFF + (size_t)i * 256 * KiB + wave * 32 * KiB);
  const u32 astep = diag ? 32 : 1024, av0 = diag ? (lane & 31) * 4096 + (lane >> 5) * 16 : voff;
  const char* fq = my + XF_OFF + (i * 64 + (lane & 7) * 8 + wave) * 128;
  char* const own = (diag ? my : peer) + slot;
  const int ysrc = diag ? j : r, yi = diag ? r : i;
  u32x4 A[32];
  u32 rdy = diag ? 8 : 0;
#define XWAIT(q) do { while (rdy <= (u32)(q)) { \
      u32 v_ = 0; if ((u32)lane - rdy < (u32)LOOKA && lane < 8) v_ = ld_flag(fq); \
      rdy += __builtin_ctz(~(u32)(__ballot(v_ >= call) >> rdy)); \
      if (rdy <= (u32)(q) && expired(my, t0, it, call)) { lds_st(C_FAIL, 1); rdy = 8; } } } while (0)
#pragma unroll
  for (int q = 0; q < 8; ++q) {
    XWAIT(q);
    if (wave == 0) { if (q == 0) STAMP(4); if (q == 3) STAMP(5); if (q == 7) STAMP(7); }
    load4v(A + q * 4, av0, av0 + astep, av0 + 2 * astep, av0 + 3 * astep, abase + q * 4 * astep);
  }
  u32x4 yv = {};
#define SEND_Y(g, t) do { const int tl_ = (ysrc * 4 + (g)) * 8 + yi; \
      st16_wt(own + YS_OFF + (size_t)(tl_ * 8 + m * 2 + kh) * KiB + lane * 16, yv); \
      u32 n_ = 1; if (lane == 0) asm volatile("ds_add_rtn_u32 %0, %1, %0\n\ts_waitcnt lgkmcnt(0)" : "+v"(n_) : "v"(C_BD + 8 + (t) * 4) : "memory"); \
      if (__builtin_amdgcn_readfirstlane(n_) == 7 && lane < 32) \
        st16_wt(own + YE_OFF + tl_ * 512 + lane * 16, *(const u32x4*)(smem + L_EX + (t) * 512 + lane * 16)); } while (0)
  const int ntile = role == 0 ? 2 : 1;
  int gam = 0;
  bool pend = false;
  const u64 tslot = diag ? 0 : t0 + G0 + (i - (i > j)) * GSTEP / 10;
#define TRY_SEND() do { if (pend && now() >= tslot) { SEND_Y(0, 0); pend = false; if (wave == 0) STAMP(11); } } while (0)
  const char* const wfl = my + WF_OFF + ((j * 8 + (lane & 1)) * 8 + wave) * 128;
  const char* const wloc = my + slot + WS_OFF + (size_t)(j * 256 + wave * 8) * KiB;
#define WAIT_W(g, mask) do { while (true) { u32 v_ = call; if (lane < 2) v_ = ld_flag(wfl + (g) * 2048); \
      if ((~(u32)__ballot(v_ >= call) & (mask)) == 0) break; \
      TRY_SEND(); \
      if (expired<WSLEEP>(my, t0, it, call)) { lds_st(C_FAIL, 1); break; } } } while (0)
#define DMA_W(g, c, t) dma4(voff, uni(wloc + (g) * 64 * KiB + (c) * 4 * KiB), lds0 + L_B + (t) * 65536 + wave * 8192 + (c) * 4096)
#define PUBLISH(t, bits) do { if (lane == 0) asm volatile("ds_add_u32 %0, %1" :: "v"(C_BD + (t) * 4), "v"((bits) << wave) : "memory"); } while (0)
#define GATE(g) do { const u64 tg_ = tslot + (g) * 7 * GSTEP / 10; while (now() < tg_) __builtin_amdgcn_s_sleep(1); } while (0)
#pragma unroll 1
  for (int tile = 0; tile < ntile; ++tile) {
    const int C_T = C_BD + tile * 4;
    gam = role == 0 ? 2 * tile : 2 * role - 1;
    if (diag) {
      TRY_SEND();
      const char* src = uni((const char*)p.w + (size_t)((j >> 2) * 1024 + wave * 128) * 512 + ((j >> 1) & 1) * 256 + gam * 64);
      const u32 lb = lds0 + L_B + tile * 65536 + wave * 8192;
#pragma unroll
      for (int t = 0; t < 8; ++t) dma1((lane >> 2) * 512 + (lane & 3) * 16, src + t * 8192, lb + t * 1024);
      vmwait<0>();
      const u32 ta = lb + ((lane >> 5) * 8 + ((lane & 15) >> 2)) * 64 + ((lane >> 4) & 1) * 32 + (lane & 3) * 8;
      u32x2 e[16];
      asm volatile("ds_read_b64_tr_b16 %0, %16\n\tds_read_b64_tr_b16 %1, %16 offset:256\n\t"
                   "ds_read_b64_tr_b16 %2, %16 offset:1024\n\tds_read_b64_tr_b16 %3, %16 offset:1280\n\t"
                   "ds_read_b64_tr_b16 %4, %16 offset:2048\n\tds_read_b64_tr_b16 %5, %16 offset:2304\n\t"
                   "ds_read_b64_tr_b16 %6, %16 offset:3072\n\tds_read_b64_tr_b16 %7, %16 offset:3328\n\t"
                   "ds_read_b64_tr_b16 %8, %16 offset:4096\n\tds_read_b64_tr_b16 %9, %16 offset:4352\n\t"
                   "ds_read_b64_tr_b16 %10, %16 offset:5120\n\tds_read_b64_tr_b16 %11, %16 offset:5376\n\t"
                   "ds_read_b64_tr_b16 %12, %16 offset:6144\n\tds_read_b64_tr_b16 %13, %16 offset:6400\n\t"
                   "ds_read_b64_tr_b16 %14, %16 offset:7168\n\tds_read_b64_tr_b16 %15, %16 offset:7424\n\t"
                   "s_waitcnt lgkmcnt(0)"
                   : "=&v"(e[0]), "=&v"(e[1]), "=&v"(e[2]), "=&v"(e[3]), "=&v"(e[4]), "=&v"(e[5]), "=&v"(e[6]), "=&v"(e[7]),
                     "=&v"(e[8]), "=&v"(e[9]), "=&v"(e[10]), "=&v"(e[11]), "=&v"(e[12]), "=&v"(e[13]), "=&v"(e[14]), "=&v"(e[15])
                   : "v"(ta) : "memory");
#pragma unroll
      for (int f = 0; f < 8; ++f)
        *(u32x4*)(smem + L_B + tile * 65536 + (wave * 8 + f) * 1024 + lane * 16) = u32x4{e[2 * f][0], e[2 * f][1], e[2 * f + 1][0], e[2 * f + 1][1]};
      PUBLISH(tile, 1u);
    } else {
      WAIT_W(gam, 1u);
      DMA_W(gam, 0, tile);
      if (wave == 0) { if (tile == 0) STAMP(8); else STAMP(9); }
      vmwait<0>();
      PUBLISH(tile, 1u);
      WAIT_W(gam, 2u);
      DMA_W(gam, 1, tile);
      if (wave == 0) { if (tile == 0) STAMP(25); else STAMP(3); }
    }
#pragma unroll
    for (int q = 0; q < 8; ++q)
      asm volatile("" : "+v"(A[q * 4 + 0]), "+v"(A[q * 4 + 1]), "+v"(A[q * 4 + 2]), "+v"(A[q * 4 + 3]) :: "memory");
    f32x16 acc = {};
    const char* bl = smem + L_B + tile * 65536 + kh * 32 * 1024 + lane * 16;
    u32 st = 0;
#define BWAIT(mask) do { while ((st & ((mask) << (kh * 4))) != ((mask) << (kh * 4))) st = lds_ld(C_T); } while (0)
#pragma unroll
    for (int c = 0; c < 2; ++c) {
      if (c == 1) {
        vmwait<0>();
        PUBLISH(tile, 256u);
        if (wave == 0) { if (tile == 0) STAMP(26); else STAMP(6); }
        TRY_SEND();
      }
      BWAIT(15u << (c * 8));
      bf16x8 wf[8];
#pragma unroll
      for (int s = 0; s < 8; ++s) wf[s] = *(const bf16x8*)(bl + ((s >> 2) * 8 + c * 4 + (s & 3)) * 1024);
#pragma unroll
      for (int s = 0; s < 16; ++s) {
        __builtin_amdgcn_sched_barrier(0);
        if (s == 8 && c == 1) TRY_SEND();
        acc = __builtin_amdgcn_mfma_f32_32x32x16_bf16(wf[s & 7], __builtin_bit_cast(bf16x8, A[(s >> 2) * 8 + c * 4 + (s & 3)]), acc, 0, 0, 0);
        if (s < 8) wf[s] = *(const bf16x8*)(bl + (16 + (s >> 2) * 8 + c * 4 + (s & 3)) * 1024);
      }
      __builtin_amdgcn_sched_barrier(0);
    }
    if (wave == 0) { if (tile == 0) STAMP(10); else STAMP(12); }
    if (wave == 7) { if (tile == 1) STAMP(20); else STAMP(21); }
    if (pend) { GATE(0); SEND_Y(0, 0); if (wave == 0) STAMP(11); }
    f32x4 o0, o1, k0, k1;
    if (kh == 0) { k0 = acc.s0123; k1 = acc.s4567; o0 = acc.s89ab; o1 = acc.scdef; }
    else         { o0 = acc.s0123; o1 = acc.s4567; k0 = acc.s89ab; k1 = acc.scdef; }
    char* red = smem + (tile == 0 ? L_RED : L_B);
    *(f32x4*)(red + wave * 2048 + lane * 16) = o0;
    *(f32x4*)(red + wave * 2048 + 1024 + lane * 16) = o1;
    BARRIER();
    if (wave == 0) { if (tile == 0) STAMP(22); else STAMP(23); }
    const u32 failed = *(const u32*)(smem + L_CTL);
    k0 += *(const f32x4*)(red + (wave ^ 4) * 2048 + lane * 16);
    k1 += *(const f32x4*)(red + (wave ^ 4) * 2048 + 1024 + lane * 16);
    u32 d0 = pack2(k0[0], k0[1]), d1 = pack2(k0[2], k0[3]), d2 = pack2(k1[0], k1[1]), d3 = pack2(k1[2], k1[3]);
    auto s0 = __builtin_amdgcn_permlane32_swap(d0, d2, false, false);
    auto s1 = __builtin_amdgcn_permlane32_swap(d1, d3, false, false);
    yv = u32x4{s0[0], s1[0], s0[1], s1[1]};
    if (failed) yv = u32x4{0x7fc07fc0u, 0x7fc07fc0u, 0x7fc07fc0u, 0x7fc07fc0u};
    if ((lane & 3) == 3) {
      *(u32*)(smem + L_EX + tile * 512 + (m * 2 + kh) * 64 + (lane >> 2) * 4) = (yv[3] >> 16) | tag;
      yv[3] = (yv[3] & 0xffffu) | tag;
    }
    pend = tile + 1 < ntile;
  }
  GATE(gam);
  SEND_Y(gam, ntile - 1);
  if (wave == 0) STAMP(13);
  if (tid == 0) __hip_atomic_store((gu32*)cntp, call, RLX_AGT);
  STAMP_FLUSH();
}

extern "C" void kernel_launch(int rank, int nranks, ncclComm_t comm, hipStream_t stream,
                              void* const* d_in, const int* in_sizes, int n_in,
                              void* d_out, int out_size, const int* in_dtypes, int out_dtype,
                              void* const* d_out_all, void* const* d_ws_all, size_t ws_bytes) {
  if (nranks != NR || n_in < 2 || ws_bytes < WS_NEED || (unsigned)rank >= (unsigned)NR ||
      in_sizes[0] != 256 * 2048 || in_sizes[1] != 2048 * 256 || out_size != 2048 * 256) {
    fprintf(stderr, "kernel_launch: unexpected problem shape\n");
    return;
  }
  static bool attr_done = false;
  if (!attr_done) {
    HIP_CHECK(hipFuncSetAttribute((const void*)ag_gemm_kernel, hipFuncAttributeMaxDynamicSharedMemorySize, L_BYTES));
    attr_done = true;
  }
  Params p;
  p.x = (const uint16_t*)d_in[0];
  p.w = (const uint16_t*)d_in[1];
  p.out = (uint16_t*)d_out;
  p.my = (char*)d_ws_all[rank];
  for (int q = 0; q < NR; ++q) p.ws[q] = (char*)d_ws_all[q];
  p.rank = rank;
  ag_gemm_kernel<<<256, 512, L_BYTES, stream>>>(p);
  HIP_CHECK(hipGetLastError());
}
```

```cpp
#include <hip/hip_runtime.h>
#include <rccl/rccl.h>
#include <cstdint>
#include <cstdio>

#define HIP_CHECK(expr) do { hipError_t _e = (expr); if (_e != hipSuccess) { \
    fprintf(stderr, "HIP error %s at %s:%d\n", hipGetErrorString(_e), __FILE__, __LINE__); \
    return; } } while (0)

typedef unsigned int u32;
typedef unsigned long long u64;
typedef u32 u32x2 __attribute__((ext_vector_type(2)));
typedef u32 u32x4 __attribute__((ext_vector_type(4)));
typedef float f32x4 __attribute__((ext_vector_type(4)));
typedef float f32x16 __attribute__((ext_vector_type(16)));
typedef __bf16 bf16x8 __attribute__((ext_vector_type(8)));
typedef __bf16 bf16x2 __attribute__((ext_vector_type(2)));
typedef __attribute__((address_space(1))) u32 gu32;
typedef __attribute__((address_space(3))) u32 lu32;

constexpr int NR = 8;
constexpr size_t KiB = 1024, MiB = 1024 * 1024;
constexpr size_t PS = 7 * MiB;
constexpr size_t XS_OFF = 0;
constexpr size_t WS_OFF = 2 * MiB;
constexpr size_t YS_OFF = 4 * MiB;
constexpr size_t YE_OFF = 6 * MiB;
constexpr size_t CTRL = 2 * PS;
constexpr size_t XF_OFF = CTRL;
constexpr size_t WF_OFF = CTRL + 64 * KiB;
constexpr size_t CNT_OFF = CTRL + 128 * KiB;
constexpr size_t TMO_OFF = CTRL + 160 * KiB;
constexpr size_t DBG_OFF = CTRL + 256 * KiB;
constexpr size_t WS_NEED = CTRL + 512 * KiB;
constexpr u64 DEADLINE = 50000000ull;
constexpr int L_B = 0, L_RED = 128 * 1024, L_CTL = 144 * 1024, L_EX = L_CTL + 512, L_BYTES = L_EX + 1024;

struct Params {
  const uint16_t* x;
  const uint16_t* w;
  uint16_t* out;
  char* my;
  char* ws[NR];
  int rank;
  u32 dest;
};

#define RLX_SYS __ATOMIC_RELAXED, __HIP_MEMORY_SCOPE_SYSTEM
#define RLX_AGT __ATOMIC_RELAXED, __HIP_MEMORY_SCOPE_AGENT
#define BARRIER() asm volatile("s_waitcnt lgkmcnt(0)\n\ts_barrier" ::: "memory")

__device__ __forceinline__ u64 now() { return __builtin_amdgcn_s_memrealtime(); }
__device__ __forceinline__ void st16_wt(void* p, u32x4 v) {
  asm volatile("global_store_dwordx4 %0, %1, off sc0 sc1\n\ts_nop 1" :: "v"(p), "v"(v) : "memory");
}
__device__ __forceinline__ void st_flag(char* p, u32 v) { __hip_atomic_store((gu32*)p, v, RLX_SYS); }
__device__ __forceinline__ u32 ld_flag(const char* p) { return __hip_atomic_load((gu32*)p, RLX_SYS); }
__device__ __forceinline__ u32 lds_ld(int off) {
  u32 v; asm volatile("ds_read_b32 %0, %1\n\ts_waitcnt lgkmcnt(0)" : "=v"(v) : "v"(off) : "memory");
  return __builtin_amdgcn_readfirstlane(v);
}
__device__ __forceinline__ void lds_st(int off, u32 v) { asm volatile("ds_write_b32 %0, %1" :: "v"(off), "v"(v) : "memory"); }
__device__ __forceinline__ const char* uni(const char* p) {
  u64 a = (u64)p;
  u32 lo = __builtin_amdgcn_readfirstlane((u32)a), hi = __builtin_amdgcn_readfirstlane((u32)(a >> 32));
  return (const char*)(((u64)hi << 32) | lo);
}
#define LSC " sc0 sc1"
__device__ __forceinline__ void load4(u32x4* A, u32 voff, const char* base) {
  asm volatile("s_nop 4\n\t"
               "global_load_dwordx4 %0, %4, %5" LSC "\n\t"
               "global_load_dwordx4 %1, %4, %5 offset:1024" LSC "\n\t"
               "global_load_dwordx4 %2, %4, %5 offset:2048" LSC "\n\t"
               "global_load_dwordx4 %3, %4, %5 offset:3072" LSC
               : "=&v"(A[0]), "=&v"(A[1]), "=&v"(A[2]), "=&v"(A[3])
               : "v"(voff), "s"(base) : "memory");
}
template <int O> __device__ __forceinline__ void loadx4(u32x4* A, u32 voff, const char* base) {
  asm volatile("s_nop 4\n\t"
               "global_load_dwordx4 %0, %4, %5 offset:%6\n\t"
               "global_load_dwordx4 %1, %4, %5 offset:%6+32\n\t"
               "global_load_dwordx4 %2, %4, %5 offset:%6+64\n\t"
               "global_load_dwordx4 %3, %4, %5 offset:%6+96"
               : "=&v"(A[0]), "=&v"(A[1]), "=&v"(A[2]), "=&v"(A[3])
               : "v"(voff), "s"(base), "i"(O) : "memory");
}
__device__ __forceinline__ void loadx32(u32x4* A, int lane, const char* base) {
  const u32 o = (lane & 31) * 4096 + (lane >> 5) * 16;
  loadx4<0>(A, o, base); loadx4<128>(A + 4, o, base); loadx4<256>(A + 8, o, base); loadx4<384>(A + 12, o, base);
  loadx4<512>(A + 16, o, base); loadx4<640>(A + 20, o, base); loadx4<768>(A + 24, o, base); loadx4<896>(A + 28, o, base);
}
__device__ __forceinline__ void dma4(u32 voff, const char* base, u32 lds) {
  u32 keep;
  asm volatile("s_mov_b32 %0, m0\n\ts_mov_b32 m0, %3\n\ts_nop 4\n\t"
               "global_load_lds_dwordx4 %1, %2" LSC "\n\t"
               "global_load_lds_dwordx4 %1, %2 offset:1024" LSC "\n\t"
               "global_load_lds_dwordx4 %1, %2 offset:2048" LSC "\n\t"
               "global_load_lds_dwordx4 %1, %2 offset:3072" LSC "\n\t"
               "s_mov_b32 m0, %0"
               : "=&s"(keep) : "v"(voff), "s"(base), "s"(lds) : "memory");
}
__device__ __forceinline__ void dma1(u32 voff, const char* base, u32 lds) {
  u32 keep;
  asm volatile("s_mov_b32 %0, m0\n\ts_mov_b32 m0, %3\n\ts_nop 4\n\t"
               "global_load_lds_dwordx4 %1, %2\n\t"
               "s_mov_b32 m0, %0"
               : "=&s"(keep) : "v"(voff), "s"(base), "s"(lds) : "memory");
}
template <int N> __device__ __forceinline__ void vmwait() { asm volatile("s_waitcnt vmcnt(%0)" :: "i"(N) : "memory"); }
template <int N> __device__ __forceinline__ u32 get_call(u32 cnt) {
  u32 c; asm volatile("s_waitcnt vmcnt(%2)\n\tv_readfirstlane_b32 %0, %1\n\ts_nop 1" : "=s"(c) : "v"(cnt), "i"(N) : "memory");
  return c + 1;
}
__device__ __forceinline__ u32 pack2(float lo, float hi) {
  bf16x2 v; v[0] = (__bf16)lo; v[1] = (__bf16)hi;
  return __builtin_bit_cast(u32, v);
}
__device__ __forceinline__ float bflo(u32 v) { return __builtin_bit_cast(float, v << 16); }
__device__ __forceinline__ float bfhi(u32 v) { return __builtin_bit_cast(float, v & 0xffff0000u); }

#define STAMP(k) do {} while (0)
#define STAMP_FLUSH() do {} while (0)

#define WIN 5
#define PSLEEP 10
#define WSLEEP 0
#define G0 985
#define GSTEP 146
#define TOPO 0x4Du
#define SLOT0 1
#define XC0 3
#define XC1 6
constexpr int xflag_of(int q) { return q <= XC0 ? 0 : (q <= XC1 ? 1 : 2); }
#define WDMA_AT 100
#define L2WARM 1
#define LOOKA 2
#define PRE 12
static_assert(PRE > WIN && PRE <= 32 && PRE % 4 == 0, "");
template <int O> __device__ __forceinline__ void loadx1(u32x4& A, u32 voff, const char* base) {
  asm volatile("global_load_dwordx4 %0, %1, %2 offset:%3" : "=&v"(A) : "v"(voff), "s"(base), "i"(O) : "memory");
}
constexpr bool push_flagged(int t) { return t < 32 ? t == 4 * XC0 + 3 || t == 4 * XC1 + 3 || t == 31 : (t & 3) == 3; }
constexpr int push_younger(int t) {
  int n = (t + WIN - 1 < 63 ? t + WIN - 1 : 63) - t;
  for (int T = t + 1; T < t + WIN; ++T) if (T >= WIN && push_flagged(T - WIN)) ++n;
  for (int T = t; T < t + WIN; ++T) if (T + PRE < 32) ++n;
  return n;
}
template <int T>
__device__ __forceinline__ void push_steps(u32x4* v, u32 xo, const char* src, char* dx, char* dw, char* fx, char* fw, u32 ta, u32 call, int lane) {
  if constexpr (T < 64 + WIN) {
    if constexpr (T >= WIN) {
      constexpr int t = T - WIN;
      if constexpr (T < 64 || push_flagged(t)) vmwait<push_younger(t)>();
      if constexpr (push_flagged(t)) {
        if (lane == 0) st_flag(t < 32 ? fx + xflag_of(t >> 2) * 1024 : fw + ((t - 32) >> 2) * 1024, call);
        if (t == 4 * XC0 + 3) STAMP(16); if (t == 31) STAMP(17); if (t == 39) STAMP(18);
      }
    } else {
      vmwait<PRE - 1 + (PRE < 32 ? T : 0)>();
    }
    if constexpr (T < 32) { asm volatile("" : "+v"(v[T])); st16_wt(dx + T * KiB, v[T]); if (T == 0) STAMP(1); }
    else if constexpr (T < 64) st16_wt(dw + (((T - 32) >> 3) * 64 + (T & 7)) * KiB, v[T - 32]);
    if constexpr (T + PRE < 32) loadx1<(T + PRE) * 32>(v[T + PRE], xo, src);
    if constexpr (T == 16) BARRIER();
    if constexpr (T >= 16 && T < 32) {
      constexpr int f = 2 * (T - 16), off = (f & 7) * 4096 + (f >> 3) * 64;
      u32x2 a0, a1, a2, a3;
      asm volatile("ds_read_b64_tr_b16 %0, %4 offset:%5\n\t"
                   "ds_read_b64_tr_b16 %1, %4 offset:%5+1024\n\t"
                   "ds_read_b64_tr_b16 %2, %4 offset:%5+4096\n\t"
                   "ds_read_b64_tr_b16 %3, %4 offset:%5+5120\n\t"
                   "s_waitcnt lgkmcnt(0)"
                   : "=&v"(a0), "=&v"(a1), "=&v"(a2), "=&v"(a3) : "v"(ta), "i"(off) : "memory");
      v[f] = u32x4{a0[0], a0[1], a1[0], a1[1]};
      v[f + 1] = u32x4{a2[0], a2[1], a3[0], a3[1]};
    }
    push_steps<T + 1>(v, xo, src, dx, dw, fx, fw, ta, call, lane);
  }
}
template <int SL = PSLEEP>
__device__ __forceinline__ bool expired(char* my, u64 t0, u32& it, u32 call) {
  if ((++it & 63) == 0 && (ld_flag(my + TMO_OFF) == call || now() - t0 > DEADLINE)) {
    __hip_atomic_store((gu32*)(my + TMO_OFF), call, RLX_AGT);
    return true;
  }
  if (SL) __builtin_amdgcn_s_sleep(SL);
  return false;
}

__global__ __launch_bounds__(512, 2) void ag_gemm_kernel(Params p) {
  extern __shared__ __attribute__((aligned(16))) char smem[];
  const int b = blockIdx.x, tid = threadIdx.x, lane = tid & 63;
  const int wave = __builtin_amdgcn_readfirstlane(tid >> 6);
  const int r = p.rank & 7;
  const int xc = (p.dest >> (4 * (b & 7))) & 7, u8 = (b >> 3) & 7, role = (3 + (b >> 6)) & 3;
  char* const my = p.my;
  char* const peer = p.ws[b & 7];
  const char* xin = (const char*)p.x;
  asm volatile("" : "+s"(xin));
  const u32 lds0 = (u32)(size_t)(lu32*)smem;
  const int C_FAIL = lds0 + L_CTL, C_BD = C_FAIL + 16;
  const u32 voff = lane * 16;
  const u64 t0 = now();
  char* const cntp = my + CNT_OFF + b * 128;
  u32 cnt;
  asm volatile("global_load_dword %0, %1, off sc1" : "=v"(cnt) : "v"(cntp) : "memory");
  u32 call;

  if (__builtin_expect(role == 3 && wave == 0, 1)) {
    const int h = xc, sub = u8;
    const char* src = uni(xin + (size_t)((h & 1) * 128 + (sub & 3) * 32) * 4096 + ((h >> 2) * 1024 + (sub >> 2) * 512) * 2);
    u32x4 v[32];
    const u32 xo = (lane & 31) * 4096 + (lane >> 5) * 16;
    loadx4<0>(v, xo, src); loadx4<128>(v + 4, xo, src);
    if (PRE > 8) loadx4<256>(v + 8, xo, src);
    if (PRE > 12) loadx4<384>(v + 12, xo, src);
    if (PRE > 16) loadx4<512>(v + 16, xo, src);
    if (PRE > 20) loadx4<640>(v + 20, xo, src);
    if (PRE > 24) loadx4<768>(v + 24, xo, src);
    if (PRE > 28) loadx4<896>(v + 28, xo, src);
    call = get_call<PRE>(cnt);
    char* base = peer + (call & 1) * PS + (size_t)r * 256 * KiB + lane * 16;
    push_steps<0>(v, xo, src, base + XS_OFF + sub * 32 * KiB, base + WS_OFF + sub * 8 * KiB, peer + XF_OFF + (r * 64 + sub) * 128, peer + WF_OFF + (r * 64 + sub) * 128,
                  lds0 + ((lane >> 5) * 8 + ((lane & 15) >> 2)) * 256 + ((lane >> 4) & 1) * 32 + (lane & 3) * 8, call, lane);
    if (lane == 0) __hip_atomic_store((gu32*)cntp, call, RLX_AGT);
    STAMP(2);
    vmwait<0>();
    BARRIER();
  } else {
    if (wave == 1 && lane < 32) lds_st(C_FAIL + lane * 4, 0);
    if (role == 3) {
      if (wave == 1) {
        const char* src = uni((const char*)p.w + (size_t)((xc >> 2) * 1024 + u8 * 128) * 512 + ((xc >> 1) & 1) * 256);
        const u32 soff = (lane >> 4) * 512 + (lane & 15) * 16;
        while (now() < t0 + WDMA_AT) __builtin_amdgcn_s_sleep(4);
#pragma unroll
        for (int t = 0; t < 32; ++t) dma1(soff, src + t * 2048, lds0 + t * 1024);
      }
      call = get_call<0>(cnt);
      BARRIER();
      BARRIER();
    } else {
      if (role == 1) {
        const char* src = uni(xin + (size_t)((xc & 1) * 128 + u8 * 16 + wave * 2) * 4096 + (xc >> 2) * 2048);
        dma1(voff, src, lds0 + L_RED + wave * 1024); dma1(voff, src + 1024, lds0 + L_RED + wave * 1024);
        dma1(voff, src + 4096, lds0 + L_RED + wave * 1024); dma1(voff, src + 5120, lds0 + L_RED + wave * 1024);
      } else if (role == 2) {
        const char* src = uni((const char*)p.w + (size_t)((xc >> 2) * 1024 + u8 * 128 + wave * 16) * 512 + ((xc >> 1) & 1) * 256);
#pragma unroll
        for (int t = 0; t < 4; ++t) dma1((lane >> 4) * 512 + (lane & 15) * 16, src + t * 2048, lds0 + L_RED + wave * 1024);
      }
      BARRIER();
      call = get_call<0>(cnt);
    }
  }
  const size_t slot = (call & 1) * PS;
  const u32 tag = (call % 65535u + 1u) << 16;
  u32 it = 0;

  if (role == 3) {
    const int u = xc * 8 + u8, ua = u & 1, ub = (u >> 1) & 1, par = (u >> 2) & 1, i2 = u >> 3;
    const int m2 = wave >> 1, kh2 = wave & 1;
    const bool tl = (lane & 3) == 3;
#pragma unroll 1
    for (int tile = 0; tile < 2; ++tile) {
      const int gam = par + 2 * tile;
      const int unit0 = (((ua + 2 * ub) * 4 + gam) * 8 + i2) * 8 + wave, unit1 = unit0 + 4 * 256;
      const char* e0 = my + slot + YE_OFF + unit0 * 64 + (lane >> 2) * 4;
      const char* e1 = e0 + 4 * 256 * 64;
      const char* y0p = my + slot + YS_OFF + (size_t)unit0 * KiB + lane * 16;
      const char* y1p = my + slot + YS_OFF + (size_t)unit1 * KiB + lane * 16;
      u32x4 o = {0x7fc07fc0u, 0x7fc07fc0u, 0x7fc07fc0u, 0x7fc07fc0u};
      while (tile == 0) {
        u32 v_ = 0; if (lane < 2) v_ = ld_flag(e0 + lane * 4 * 256 * 64);
        if (__ballot((v_ ^ tag) >> 16 == 0) || expired<WSLEEP>(my, t0, it, call)) break;
      }
      while (true) {
        u32x4 y0, y1; u32 x0, x1, tm;
        asm volatile("global_load_dwordx4 %0, %5, off sc0 sc1\n\tglobal_load_dwordx4 %1, %6, off sc0 sc1\n\t"
                     "global_load_dword %2, %7, off sc0 sc1\n\tglobal_load_dword %3, %8, off sc0 sc1\n\t"
                     "global_load_dword %4, %9, off sc1\n\ts_waitcnt vmcnt(0)"
                     : "=&v"(y0), "=&v"(y1), "=&v"(x0), "=&v"(x1), "=&v"(tm) : "v"(y0p), "v"(y1p), "v"(e0), "v"(e1), "v"(my + TMO_OFF) : "memory");
        if (__ballot(tm == call)) break;
        if (__ballot(!tl || ((y0[3] ^ tag) | (y1[3] ^ tag) | (x0 ^ tag) | (x1 ^ tag)) >> 16 == 0) == ~0ull) {
          if (wave == 2) { if (tile == 0) STAMP(19); else STAMP(14); }
          if (tl) { y0[3] = (y0[3] & 0xffffu) | (x0 << 16); y1[3] = (y1[3] & 0xffffu) | (x1 << 16); }
#pragma unroll
          for (int t = 0; t < 4; ++t) o[t] = pack2(bflo(y0[t]) + bflo(y1[t]), bfhi(y0[t]) + bfhi(y1[t]));
          break;
        }
        if (expired<WSLEEP>(my, t0, it, call)) break;
      }
      *(u32x4*)(p.out + (size_t)(i2 * 256 + ua * 128 + m2 * 32 + (lane & 31)) * 256 + ub * 128 + gam * 32 + kh2 * 16 + (lane >> 5) * 8) = o;
    }
    STAMP(15);
    STAMP_FLUSH();
    return;
  }

  const int i = u8, j = xc;
  const int m = wave & 3, kh = wave >> 2;
  const bool diag = i == j;
  const char* abase = uni(diag ? xin + (size_t)((j & 1) * 128 + m * 32) * 4096 + ((j >> 2) * 1024 + kh * 512) * 2
                               : my + slot + XS_OFF + (size_t)i * 256 * KiB + wave * 32 * KiB);
  const char* fq = my + XF_OFF + (i * 64 + (lane & 7) * 8 + wave) * 128;
  char* const own = (diag ? my : peer) + slot;
  const int ysrc = diag ? j : r, yi = diag ? r : i;
  u32x4 A[32];
  u32 rdy = 0;
#define XWAIT(q) do { while (rdy <= (u32)(q)) { \
      u32 v_ = 0; if ((u32)lane - rdy < (u32)LOOKA && lane < 3) v_ = ld_flag(fq); \
      rdy += __builtin_ctz(~(u32)(__ballot(v_ >= call) >> rdy)); \
      if (rdy <= (u32)(q) && expired(my, t0, it, call)) { lds_st(C_FAIL, 1); rdy = 8; } } } while (0)
  if (diag) {
    loadx32(A, lane, abase);
  } else {
#pragma unroll
    for (int q = 0; q < 8; ++q) {
      XWAIT(xflag_of(q));
      if (wave == 0) { if (q == 0) STAMP(4); if (q == 3) STAMP(5); if (q == 7) STAMP(7); }
      load4(A + q * 4, voff, abase + q * 4 * KiB);
    }
  }
  u32x4 yv = {}, y0v = {};
#define SEND_Y(g, t, yv) do { const int tl_ = (ysrc * 4 + (g)) * 8 + yi; \
      st16_wt(own + YS_OFF + (size_t)(tl_ * 8 + m * 2 + kh) * KiB + lane * 16, yv); \
      u32 n_ = 1; if (lane == 0) asm volatile("ds_add_rtn_u32 %0, %1, %0\n\ts_waitcnt lgkmcnt(0)" : "+v"(n_) : "v"(C_BD + 8 + (t) * 4) : "memory"); \
      if (__builtin_amdgcn_readfirstlane(n_) == 7 && lane < 32) \
        st16_wt(own + YE_OFF + tl_ * 512 + lane * 16, *(const u32x4*)(smem + L_EX + (t) * 512 + lane * 16)); } while (0)
  const int ntile = role == 0 ? 2 : 1;
  int gam = 0;
  u64 tn = 0;
  bool pend = false;
  const u32 tpos = i - (i > j);
  u64 tg = 0, tg0 = 0;
#define TRY_SEND() do { if (pend && now() >= tg0) { SEND_Y(0, 0, y0v); pend = false; if (wave == 0) STAMP(11); } } while (0)
  const char* const wfl = my + WF_OFF + ((j * 8 + (lane & 1)) * 8 + wave) * 128;
  const char* const wloc = my + slot + WS_OFF + (size_t)(j * 256 + wave * 8) * KiB;
#define WAIT_W(g, mask) do { while (true) { u32 v_ = call; if (lane < 2) v_ = ld_flag(wfl + (g) * 2048); \
      if ((~(u32)__ballot(v_ >= call) & (mask)) == 0) break; \
      if ((mask) == 1u) TRY_SEND(); \
      if (expired<WSLEEP>(my, t0, it, call)) { lds_st(C_FAIL, 1); break; } } } while (0)
#define DMA_W(g, c, t) dma4(voff, uni(wloc + (g) * 64 * KiB + (c) * 4 * KiB), lds0 + L_B + (t) * 65536 + wave * 8192 + (c) * 4096)
#define PUBLISH(t, bits) do { if (lane == 0) asm volatile("ds_add_u32 %0, %1" :: "v"(C_BD + (t) * 4), "v"((bits) << wave) : "memory"); } while (0)
#define GATE(tg_) do { if (tn < (tg_)) while (now() < (tg_)) __builtin_amdgcn_s_sleep(1); } while (0)
#pragma unroll 1
  for (int tile = 0; tile < ntile; ++tile) {
    const int C_T = C_BD + tile * 4;
    gam = role == 0 ? 2 * tile : 2 * role - 1;
    TRY_SEND();
    if (diag) {
      const char* src = uni((const char*)p.w + (size_t)((j >> 2) * 1024 + wave * 128) * 512 + ((j >> 1) & 1) * 256 + gam * 64);
      const u32 lb = lds0 + L_B + tile * 65536 + wave * 8192;
#pragma unroll
      for (int t = 0; t < 8; ++t) dma1((lane >> 2) * 512 + (lane & 3) * 16, src + t * 8192, lb + t * 1024);
      vmwait<0>();
      const u32 ta = lb + ((lane >> 5) * 8 + ((lane & 15) >> 2)) * 64 + ((lane >> 4) & 1) * 32 + (lane & 3) * 8;
      u32x2 e[16];
      asm volatile("ds_read_b64_tr_b16 %0, %16\n\tds_read_b64_tr_b16 %1, %16 offset:256\n\t"
                   "ds_read_b64_tr_b16 %2, %16 offset:1024\n\tds_read_b64_tr_b16 %3, %16 offset:1280\n\t"
                   "ds_read_b64_tr_b16 %4, %16 offset:2048\n\tds_read_b64_tr_b16 %5, %16 offset:2304\n\t"
                   "ds_read_b64_tr_b16 %6, %16 offset:3072\n\tds_read_b64_tr_b16 %7, %16 offset:3328\n\t"
                   "ds_read_b64_tr_b16 %8, %16 offset:4096\n\tds_read_b64_tr_b16 %9, %16 offset:4352\n\t"
                   "ds_read_b64_tr_b16 %10, %16 offset:5120\n\tds_read_b64_tr_b16 %11, %16 offset:5376\n\t"
                   "ds_read_b64_tr_b16 %12, %16 offset:6144\n\tds_read_b64_tr_b16 %13, %16 offset:6400\n\t"
                   "ds_read_b64_tr_b16 %14, %16 offset:7168\n\tds_read_b64_tr_b16 %15, %16 offset:7424\n\t"
                   "s_waitcnt lgkmcnt(0)"
                   : "=&v"(e[0]), "=&v"(e[1]), "=&v"(e[2]), "=&v"(e[3]), "=&v"(e[4]), "=&v"(e[5]), "=&v"(e[6]), "=&v"(e[7]),
                     "=&v"(e[8]), "=&v"(e[9]), "=&v"(e[10]), "=&v"(e[11]), "=&v"(e[12]), "=&v"(e[13]), "=&v"(e[14]), "=&v"(e[15])
                   : "v"(ta) : "memory");
#pragma unroll
      for (int f = 0; f < 8; ++f)
        *(u32x4*)(smem + L_B + tile * 65536 + (wave * 8 + f) * 1024 + lane * 16) = u32x4{e[2 * f][0], e[2 * f][1], e[2 * f + 1][0], e[2 * f + 1][1]};
      PUBLISH(tile, 1u);
    } else {
      WAIT_W(gam, 1u);
      DMA_W(gam, 0, tile);
      if (wave == 0) { if (tile == 0) STAMP(8); else STAMP(9); }
      vmwait<0>();
      PUBLISH(tile, 1u);
      WAIT_W(gam, 2u);
      DMA_W(gam, 1, tile);
      if (wave == 0) { if (tile == 0) STAMP(25); else STAMP(3); }
    }
#pragma unroll
    for (int q = 0; q < 8; ++q)
      asm volatile("" : "+v"(A[q * 4 + 0]), "+v"(A[q * 4 + 1]), "+v"(A[q * 4 + 2]), "+v"(A[q * 4 + 3]) :: "memory");
    f32x16 acc = {};
    const char* bl = smem + L_B + tile * 65536 + kh * 32 * 1024 + lane * 16;
    u32 st = 0;
#define BWAIT(mask) do { while ((st & ((mask) << (kh * 4))) != ((mask) << (kh * 4))) st = lds_ld(C_T); } while (0)
#pragma unroll
    for (int c = 0; c < 2; ++c) {
      if (c == 1) {
        vmwait<0>();
        PUBLISH(tile, 256u);
        if (wave == 0) { if (tile == 0) STAMP(26); else STAMP(6); }
        TRY_SEND();
      }
      BWAIT(15u << (c * 8));
      bf16x8 wf[8];
#pragma unroll
      for (int s = 0; s < 8; ++s) wf[s] = *(const bf16x8*)(bl + ((s >> 2) * 8 + c * 4 + (s & 3)) * 1024);
#pragma unroll
      for (int s = 0; s < 16; ++s) {
        __builtin_amdgcn_sched_barrier(0);
        if (s == 8 && c == 1) TRY_SEND();
        acc = __builtin_amdgcn_mfma_f32_32x32x16_bf16(wf[s & 7], __builtin_bit_cast(bf16x8, A[(s >> 2) * 8 + c * 4 + (s & 3)]), acc, 0, 0, 0);
        if (s < 8) wf[s] = *(const bf16x8*)(bl + (16 + (s >> 2) * 8 + c * 4 + (s & 3)) * 1024);
      }
      __builtin_amdgcn_sched_barrier(0);
    }
    if (wave == 0) { if (tile == 0) STAMP(10); else STAMP(12); }
    if (wave == 7) { if (tile == 1) STAMP(20); else STAMP(21); }
    TRY_SEND();
    f32x4 o0, o1, k0, k1;
    if (kh == 0) { k0 = acc.s0123; k1 = acc.s4567; o0 = acc.s89ab; o1 = acc.scdef; }
    else         { o0 = acc.s0123; o1 = acc.s4567; k0 = acc.s89ab; k1 = acc.scdef; }
    char* red = smem + (tile == 0 ? L_RED : L_B);
    *(f32x4*)(red + wave * 2048 + lane * 16) = o0;
    *(f32x4*)(red + wave * 2048 + 1024 + lane * 16) = o1;
    tn = now();
    BARRIER();
    if (wave == 0) { if (tile == 0) STAMP(22); else STAMP(23); }
    const u32 failed = *(const u32*)(smem + L_CTL);
    k0 += *(const f32x4*)(red + (wave ^ 4) * 2048 + lane * 16);
    k1 += *(const f32x4*)(red + (wave ^ 4) * 2048 + 1024 + lane * 16);
    u32 d0 = pack2(k0[0], k0[1]), d1 = pack2(k0[2], k0[3]), d2 = pack2(k1[0], k1[1]), d3 = pack2(k1[2], k1[3]);
    auto s0 = __builtin_amdgcn_permlane32_swap(d0, d2, false, false);
    auto s1 = __builtin_amdgcn_permlane32_swap(d1, d3, false, false);
    yv = u32x4{s0[0], s1[0], s0[1], s1[1]};
    if (failed) yv = u32x4{0x7fc07fc0u, 0x7fc07fc0u, 0x7fc07fc0u, 0x7fc07fc0u};
    if ((lane & 3) == 3) {
      *(u32*)(smem + L_EX + tile * 512 + (m * 2 + kh) * 64 + (lane >> 2) * 4) = (yv[3] >> 16) | tag;
      yv[3] = (yv[3] & 0xffffu) | tag;
    }
    tg = t0 + G0 + ((gam < 2 ? gam ^ SLOT0 : gam) * 7 + tpos) * GSTEP / 10;
    if (tg < tn + tpos * GSTEP / 10) tg = tn + tpos * GSTEP / 10;
    if (diag) tg = 0;
    if (tile + 1 < ntile) { pend = true; y0v = yv; tg0 = tg; }
  }
  if (pend) { GATE(tg0); SEND_Y(0, 0, y0v); if (wave == 0) STAMP(11); }
  GATE(tg);
  SEND_Y(gam, ntile - 1, yv);
  if (wave == 0) STAMP(13);
  if (tid == 0) __hip_atomic_store((gu32*)cntp, call, RLX_AGT);
  STAMP_FLUSH();
}

extern "C" void kernel_launch(int rank, int nranks, ncclComm_t comm, hipStream_t stream,
                              void* const* d_in, const int* in_sizes, int n_in,
                              void* d_out, int out_size, const int* in_dtypes, int out_dtype,
                              void* const* d_out_all, void* const* d_ws_all, size_t ws_bytes) {
  if (nranks != NR || n_in < 2 || ws_bytes < WS_NEED || (unsigned)rank >= (unsigned)NR ||
      in_sizes[0] != 256 * 2048 || in_sizes[1] != 2048 * 256 || out_size != 2048 * 256) {
    fprintf(stderr, "kernel_launch: unexpected problem shape\n");
    return;
  }
  static bool attr_done = false;
  if (!attr_done) {
    HIP_CHECK(hipFuncSetAttribute((const void*)ag_gemm_kernel, hipFuncAttributeMaxDynamicSharedMemorySize, L_BYTES));
    attr_done = true;
  }
  Params p;
  p.x = (const uint16_t*)d_in[0];
  p.w = (const uint16_t*)d_in[1];
  p.out = (uint16_t*)d_out;
  p.my = (char*)d_ws_all[rank];
  const unsigned grp = TOPO, same = (grp >> rank) & 1 ? grp : ~grp & 0xffu;
  p.dest = 0;
  for (int q = 0, lo = 0, hi = 4; q < NR; ++q) {
    const int idx = TOPO ? ((same >> q) & 1 ? lo++ : hi++) : q;
    p.ws[idx] = (char*)d_ws_all[q];
    p.dest |= (u32)q << (4 * idx);
  }
  p.rank = rank;
  ag_gemm_kernel<<<256, 512, L_BYTES, stream>>>(p);
  HIP_CHECK(hipGetLastError());
}
```

```cpp
#include <hip/hip_runtime.h>
#include <rccl/rccl.h>
#include <cstdint>
#include <cstdio>

#define HIP_CHECK(expr) do { hipError_t _e = (expr); if (_e != hipSuccess) { \
    fprintf(stderr, "HIP error %s at %s:%d\n", hipGetErrorString(_e), __FILE__, __LINE__); \
    return; } } while (0)

typedef unsigned int u32;
typedef unsigned long long u64;
typedef u32 u32x2 __attribute__((ext_vector_type(2)));
typedef u32 u32x4 __attribute__((ext_vector_type(4)));
typedef float f32x4 __attribute__((ext_vector_type(4)));
typedef float f32x16 __attribute__((ext_vector_type(16)));
typedef __bf16 bf16x8 __attribute__((ext_vector_type(8)));
typedef __bf16 bf16x2 __attribute__((ext_vector_type(2)));
typedef __attribute__((address_space(1))) u32 gu32;
typedef __attribute__((address_space(3))) u32 lu32;

constexpr int NR = 8;
constexpr size_t KiB = 1024, MiB = 1024 * 1024;
constexpr size_t PS = 7 * MiB;
constexpr size_t XS_OFF = 0;
constexpr size_t WS_OFF = 2 * MiB;
constexpr size_t YS_OFF = 4 * MiB;
constexpr size_t YE_OFF = 6 * MiB;
constexpr size_t CTRL = 2 * PS;
constexpr size_t XF_OFF = CTRL;
constexpr size_t WF_OFF = CTRL + 64 * KiB;
constexpr size_t CNT_OFF = CTRL + 128 * KiB;
constexpr size_t TMO_OFF = CTRL + 160 * KiB;
constexpr size_t WS_NEED = CTRL + 192 * KiB;
constexpr u64 DEADLINE = 150000000ull;
constexpr int L_B = 0, L_RED = 128 * 1024, L_CTL = 144 * 1024, L_EX = L_CTL + 512, L_BYTES = L_EX + 1024;

struct Params {
  const uint16_t* x;
  const uint16_t* w;
  uint16_t* out;
  char* my;
  char* ws[NR];
  int rank;
  u32 dest;
};

#define RLX_SYS __ATOMIC_RELAXED, __HIP_MEMORY_SCOPE_SYSTEM
#define RLX_AGT __ATOMIC_RELAXED, __HIP_MEMORY_SCOPE_AGENT
#define BARRIER() asm volatile("s_waitcnt lgkmcnt(0)\n\ts_barrier" ::: "memory")

__device__ __forceinline__ u64 now() { return __builtin_amdgcn_s_memrealtime(); }
__device__ __forceinline__ void st16_wt(void* p, u32x4 v) {
  asm volatile("global_store_dwordx4 %0, %1, off sc0 sc1\n\ts_nop 1" :: "v"(p), "v"(v) : "memory");
}
__device__ __forceinline__ void st_flag(char* p, u32 v) { __hip_atomic_store((gu32*)p, v, RLX_SYS); }
__device__ __forceinline__ u32 ld_flag(const char* p) { return __hip_atomic_load((gu32*)p, RLX_SYS); }
__device__ __forceinline__ u32 lds_ld(int off) {
  u32 v; asm volatile("ds_read_b32 %0, %1\n\ts_waitcnt lgkmcnt(0)" : "=v"(v) : "v"(off) : "memory");
  return __builtin_amdgcn_readfirstlane(v);
}
__device__ __forceinline__ void lds_st(int off, u32 v) { asm volatile("ds_write_b32 %0, %1" :: "v"(off), "v"(v) : "memory"); }
__device__ __forceinline__ const char* uni(const char* p) {
  u64 a = (u64)p;
  u32 lo = __builtin_amdgcn_readfirstlane((u32)a), hi = __builtin_amdgcn_readfirstlane((u32)(a >> 32));
  return (const char*)(((u64)hi << 32) | lo);
}
#define LSC " sc0 sc1"
__device__ __forceinline__ void load4(u32x4* A, u32 voff, const char* base) {
  asm volatile("s_nop 4\n\t"
               "global_load_dwordx4 %0, %4, %5" LSC "\n\t"
               "global_load_dwordx4 %1, %4, %5 offset:1024" LSC "\n\t"
               "global_load_dwordx4 %2, %4, %5 offset:2048" LSC "\n\t"
               "global_load_dwordx4 %3, %4, %5 offset:3072" LSC
               : "=&v"(A[0]), "=&v"(A[1]), "=&v"(A[2]), "=&v"(A[3])
               : "v"(voff), "s"(base) : "memory");
}
template <int O> __device__ __forceinline__ void loadx4(u32x4* A, u32 voff, const char* base) {
  asm volatile("s_nop 4\n\t"
               "global_load_dwordx4 %0, %4, %5 offset:%6\n\t"
               "global_load_dwordx4 %1, %4, %5 offset:%6+32\n\t"
               "global_load_dwordx4 %2, %4, %5 offset:%6+64\n\t"
               "global_load_dwordx4 %3, %4, %5 offset:%6+96"
               : "=&v"(A[0]), "=&v"(A[1]), "=&v"(A[2]), "=&v"(A[3])
               : "v"(voff), "s"(base), "i"(O) : "memory");
}
__device__ __forceinline__ void loadx32(u32x4* A, int lane, const char* base) {
  const u32 o = (lane & 31) * 4096 + (lane >> 5) * 16;
  loadx4<0>(A, o, base); loadx4<128>(A + 4, o, base); loadx4<256>(A + 8, o, base); loadx4<384>(A + 12, o, base);
  loadx4<512>(A + 16, o, base); loadx4<640>(A + 20, o, base); loadx4<768>(A + 24, o, base); loadx4<896>(A + 28, o, base);
}
__device__ __forceinline__ void dma4(u32 voff, const char* base, u32 lds) {
  u32 keep;
  asm volatile("s_mov_b32 %0, m0\n\ts_mov_b32 m0, %3\n\ts_nop 4\n\t"
               "global_load_lds_dwordx4 %1, %2" LSC "\n\t"
               "global_load_lds_dwordx4 %1, %2 offset:1024" LSC "\n\t"
               "global_load_lds_dwordx4 %1, %2 offset:2048" LSC "\n\t"
               "global_load_lds_dwordx4 %1, %2 offset:3072" LSC "\n\t"
               "s_mov_b32 m0, %0"
               : "=&s"(keep) : "v"(voff), "s"(base), "s"(lds) : "memory");
}
__device__ __forceinline__ void dma1(u32 voff, const char* base, u32 lds) {
  u32 keep;
  asm volatile("s_mov_b32 %0, m0\n\ts_mov_b32 m0, %3\n\ts_nop 4\n\t"
               "global_load_lds_dwordx4 %1, %2\n\t"
               "s_mov_b32 m0, %0"
               : "=&s"(keep) : "v"(voff), "s"(base), "s"(lds) : "memory");
}
template <int N> __device__ __forceinline__ void vmwait() { asm volatile("s_waitcnt vmcnt(%0)" :: "i"(N) : "memory"); }
template <int N> __device__ __forceinline__ u32 get_call(u32 cnt) {
  u32 c; asm volatile("s_waitcnt vmcnt(%2)\n\tv_readfirstlane_b32 %0, %1\n\ts_nop 1" : "=s"(c) : "v"(cnt), "i"(N) : "memory");
  return c + 1;
}
__device__ __forceinline__ u32 pack2(float lo, float hi) {
  bf16x2 v; v[0] = (__bf16)lo; v[1] = (__bf16)hi;
  return __builtin_bit_cast(u32, v);
}
__device__ __forceinline__ float bflo(u32 v) { return __builtin_bit_cast(float, v << 16); }
__device__ __forceinline__ float bfhi(u32 v) { return __builtin_bit_cast(float, v & 0xffff0000u); }

constexpr int WIN = 5;
constexpr int PRE = 16;
constexpr int XC0 = 4, XC1 = 6;
constexpr int G0 = 1005;
constexpr int GSTEP = 146;
constexpr int WDMA_AT = 100;
constexpr int xflag_of(int q) { return q <= XC0 ? 0 : (q <= XC1 ? 1 : 2); }
static_assert(PRE > WIN && PRE <= 32 && PRE % 4 == 0, "");
static_assert(4 * XC0 + 3 + PRE >= 32, "no load may be younger than a store whose acknowledgement releases a flag");
template <int O> __device__ __forceinline__ void loadx1(u32x4& A, u32 voff, const char* base) {
  asm volatile("global_load_dwordx4 %0, %1, %2 offset:%3" : "=&v"(A) : "v"(voff), "s"(base), "i"(O) : "memory");
}
constexpr bool push_flagged(int t) { return t < 32 ? t == 4 * XC0 + 3 || t == 4 * XC1 + 3 || t == 31 : (t & 3) == 3; }
constexpr int push_younger(int t) {
  int n = (t + WIN - 1 < 63 ? t + WIN - 1 : 63) - t;
  for (int T = t + 1; T < t + WIN; ++T) if (T >= WIN && push_flagged(T - WIN)) ++n;
  for (int T = t; T < t + WIN; ++T) if (T + PRE < 32) ++n;
  return n;
}
template <int T>
__device__ __forceinline__ void push_steps(u32x4* v, u32 xo, const char* src, char* dx, char* dw, char* fx, char* fw, u32 ta, u32 call, int lane) {
  if constexpr (T < 64 + WIN) {
    if constexpr (T >= WIN) {
      constexpr int t = T - WIN;
      if constexpr (T < 64 || push_flagged(t)) vmwait<push_younger(t)>();
      if constexpr (push_flagged(t)) {
        if (lane == 0) st_flag(t < 32 ? fx + xflag_of(t >> 2) * 1024 : fw + ((t - 32) >> 2) * 1024, call);
      }
    } else {
      vmwait<PRE - 1 + (PRE < 32 ? T : 0)>();
    }
    if constexpr (T < 32) { asm volatile("" : "+v"(v[T])); st16_wt(dx + T * KiB, v[T]); }
    else if constexpr (T < 64) st16_wt(dw + (((T - 32) >> 3) * 64 + (T & 7)) * KiB, v[T - 32]);
    if constexpr (T + PRE < 32) loadx1<(T + PRE) * 32>(v[T + PRE], xo, src);
    if constexpr (T == 16) BARRIER();
    if constexpr (T >= 16 && T < 32) {
      constexpr int f = 2 * (T - 16), off = (f & 7) * 4096 + (f >> 3) * 64;
      u32x2 a0, a1, a2, a3;
      asm volatile("ds_read_b64_tr_b16 %0, %4 offset:%5\n\t"
                   "ds_read_b64_tr_b16 %1, %4 offset:%5+1024\n\t"
                   "ds_read_b64_tr_b16 %2, %4 offset:%5+4096\n\t"
                   "ds_read_b64_tr_b16 %3, %4 offset:%5+5120\n\t"
                   "s_waitcnt lgkmcnt(0)"
                   : "=&v"(a0), "=&v"(a1), "=&v"(a2), "=&v"(a3) : "v"(ta), "i"(off) : "memory");
      v[f] = u32x4{a0[0], a0[1], a1[0], a1[1]};
      v[f + 1] = u32x4{a2[0], a2[1], a3[0], a3[1]};
    }
    push_steps<T + 1>(v, xo, src, dx, dw, fx, fw, ta, call, lane);
  }
}
template <int SL>
__device__ __forceinline__ bool expired(char* my, u64 t0, u32& it, u32 call) {
  if ((++it & 63) == 0 && (ld_flag(my + TMO_OFF) == call || now() - t0 > DEADLINE)) {
    __hip_atomic_store((gu32*)(my + TMO_OFF), call, RLX_AGT);
    return true;
  }
  if (SL) __builtin_amdgcn_s_sleep(SL);
  return false;
}

__global__ __launch_bounds__(512, 2) void ag_gemm_kernel(Params p) {
  extern __shared__ __attribute__((aligned(16))) char smem[];
  const int b = blockIdx.x, tid = threadIdx.x, lane = tid & 63;
  const int wave = __builtin_amdgcn_readfirstlane(tid >> 6);
  const int r = p.rank & 7;
  const int xc = (p.dest >> (4 * (b & 7))) & 7, u8 = (b >> 3) & 7, role = (3 + (b >> 6)) & 3;
  char* const my = p.my;
  char* const peer = p.ws[b & 7];
  const char* xin = (const char*)p.x;
  asm volatile("" : "+s"(xin));
  const u32 lds0 = (u32)(size_t)(lu32*)smem;
  const int C_FAIL = lds0 + L_CTL, C_BD = C_FAIL + 16;
  const u32 voff = lane * 16;
  const u64 t0 = now();
  char* const cntp = my + CNT_OFF + b * 128;
  u32 cnt;
  asm volatile("global_load_dword %0, %1, off sc1" : "=v"(cnt) : "v"(cntp) : "memory");
  u32 call;

  if (__builtin_expect(role == 3 && wave == 0, 1)) {
    const int h = xc, sub = u8;
    const char* src = uni(xin + (size_t)((h & 1) * 128 + (sub & 3) * 32) * 4096 + ((h >> 2) * 1024 + (sub >> 2) * 512) * 2);
    u32x4 v[32];
    const u32 xo = (lane & 31) * 4096 + (lane >> 5) * 16;
    loadx4<0>(v, xo, src); loadx4<128>(v + 4, xo, src);
    if (PRE > 8) loadx4<256>(v + 8, xo, src);
    if (PRE > 12) loadx4<384>(v + 12, xo, src);
    if (PRE > 16) loadx4<512>(v + 16, xo, src);
    if (PRE > 20) loadx4<640>(v + 20, xo, src);
    if (PRE > 24) loadx4<768>(v + 24, xo, src);
    if (PRE > 28) loadx4<896>(v + 28, xo, src);
    call = get_call<PRE>(cnt);
    char* base = peer + (call & 1) * PS + (size_t)r * 256 * KiB + lane * 16;
    push_steps<0>(v, xo, src, base + XS_OFF + sub * 32 * KiB, base + WS_OFF + sub * 8 * KiB, peer + XF_OFF + (r * 64 + sub) * 128, peer + WF_OFF + (r * 64 + sub) * 128,
                  lds0 + ((lane >> 5) * 8 + ((lane & 15) >> 2)) * 256 + ((lane >> 4) & 1) * 32 + (lane & 3) * 8, call, lane);
    if (lane == 0) __hip_atomic_store((gu32*)cntp, call, RLX_AGT);
    vmwait<0>();
    BARRIER();
  } else {
    if (wave == 1 && lane < 32) lds_st(C_FAIL + lane * 4, 0);
    if (role == 3) {
      if (wave == 1) {
        const char* src = uni((const char*)p.w + (size_t)((xc >> 2) * 1024 + u8 * 128) * 512 + ((xc >> 1) & 1) * 256);
        const u32 soff = (lane >> 4) * 512 + (lane & 15) * 16;
        while (now() < t0 + WDMA_AT) __builtin_amdgcn_s_sleep(4);
#pragma unroll
        for (int t = 0; t < 32; ++t) dma1(soff, src + t * 2048, lds0 + t * 1024);
      }
      call = get_call<0>(cnt);
      BARRIER();
      BARRIER();
    } else {
      if (role == 1) {
        const char* src = uni(xin + (size_t)((xc & 1) * 128 + u8 * 16 + wave * 2) * 4096 + (xc >> 2) * 2048);
        dma1(voff, src, lds0 + L_RED + wave * 1024); dma1(voff, src + 1024, lds0 + L_RED + wave * 1024);
        dma1(voff, src + 4096, lds0 + L_RED + wave * 1024); dma1(voff, src + 5120, lds0 + L_RED + wave * 1024);
      } else if (role == 2) {
        const char* src = uni((const char*)p.w + (size_t)((xc >> 2) * 1024 + u8 * 128 + wave * 16) * 512 + ((xc >> 1) & 1) * 256);
#pragma unroll
        for (int t = 0; t < 4; ++t) dma1((lane >> 4) * 512 + (lane & 15) * 16, src + t * 2048, lds0 + L_RED + wave * 1024);
      }
      BARRIER();
      call = get_call<0>(cnt);
    }
  }
  const size_t slot = (call & 1) * PS;
  const u32 tag = (call % 65535u + 1u) << 16;
  u32 it = 0;

  if (role == 3) {
    const int u = xc * 8 + u8, ua = u & 1, ub = (u >> 1) & 1, par = (u >> 2) & 1, i2 = u >> 3;
    const int m2 = wave >> 1, kh2 = wave & 1;
    const bool tl = (lane & 3) == 3;
#pragma unroll 1
    for (int tile = 0; tile < 2; ++tile) {
      const int gam = par + 2 * tile;
      const int unit0 = (((ua + 2 * ub) * 4 + gam) * 8 + i2) * 8 + wave, unit1 = unit0 + 4 * 256;
      const char* e0 = my + slot + YE_OFF + unit0 * 64 + (lane >> 2) * 4;
      const char* e1 = e0 + 4 * 256 * 64;
      const char* y0p = my + slot + YS_OFF + (size_t)unit0 * KiB + lane * 16;
      const char* y1p = my + slot + YS_OFF + (size_t)unit1 * KiB + lane * 16;
      u32x4 o = {0x7fc07fc0u, 0x7fc07fc0u, 0x7fc07fc0u, 0x7fc07fc0u};
      while (tile == 0) {
        u32 v_ = 0; if (lane < 2) v_ = ld_flag(e0 + lane * 4 * 256 * 64);
        if (__ballot((v_ ^ tag) >> 16 == 0) || expired<0>(my, t0, it, call)) break;
      }
      while (true) {
        u32x4 y0, y1; u32 x0, x1, tm;
        asm volatile("global_load_dwordx4 %0, %5, off sc0 sc1\n\tglobal_load_dwordx4 %1, %6, off sc0 sc1\n\t"
                     "global_load_dword %2, %7, off sc0 sc1\n\tglobal_load_dword %3, %8, off sc0 sc1\n\t"
                     "global_load_dword %4, %9, off sc1\n\ts_waitcnt vmcnt(0)"
                     : "=&v"(y0), "=&v"(y1), "=&v"(x0), "=&v"(x1), "=&v"(tm) : "v"(y0p), "v"(y1p), "v"(e0), "v"(e1), "v"(my + TMO_OFF) : "memory");
        if (__ballot(tm == call)) break;
        if (__ballot(!tl || ((y0[3] ^ tag) | (y1[3] ^ tag) | (x0 ^ tag) | (x1 ^ tag)) >> 16 == 0) == ~0ull) {
          if (tl) { y0[3] = (y0[3] & 0xffffu) | (x0 << 16); y1[3] = (y1[3] & 0xffffu) | (x1 << 16); }
#pragma unroll
          for (int t = 0; t < 4; ++t) o[t] = pack2(bflo(y0[t]) + bflo(y1[t]), bfhi(y0[t]) + bfhi(y1[t]));
          break;
        }
        if (expired<0>(my, t0, it, call)) break;
      }
      *(u32x4*)(p.out + (size_t)(i2 * 256 + ua * 128 + m2 * 32 + (lane & 31)) * 256 + ub * 128 + gam * 32 + kh2 * 16 + (lane >> 5) * 8) = o;
    }
    return;
  }

  const int i = u8, j = xc;
  const int m = wave & 3, kh = wave >> 2;
  const bool diag = i == j;
  const char* abase = uni(diag ? xin + (size_t)((j & 1) * 128 + m * 32) * 4096 + ((j >> 2) * 1024 + kh * 512) * 2
                               : my + slot + XS_OFF + (size_t)i * 256 * KiB + wave * 32 * KiB);
  const char* fq = my + XF_OFF + (i * 64 + (lane & 7) * 8 + wave) * 128;
  char* const own = (diag ? my : peer) + slot;
  const int ysrc = diag ? j : r, yi = diag ? r : i;
  u32x4 A[32];
  u32 rdy = 0;
#define XWAIT(q) do { while (rdy <= (u32)(q)) { \
      u32 v_ = 0; if ((u32)lane - rdy < 2u && lane < 3) v_ = ld_flag(fq); \
      rdy += __builtin_ctz(~(u32)(__ballot(v_ >= call) >> rdy)); \
      if (rdy <= (u32)(q) && expired<10>(my, t0, it, call)) { lds_st(C_FAIL, 1); rdy = 8; } } } while (0)
#define PUBLISH(t, bits) do { if (lane == 0) asm volatile("ds_add_u32 %0, %1" :: "v"(C_BD + (t) * 4), "v"((bits) << wave) : "memory"); } while (0)
  const int ntile = role == 0 ? 2 : 1;
  if (diag) {
    for (int tile = 0; tile < ntile; ++tile) {
      const char* src = uni((const char*)p.w + (size_t)((j >> 2) * 1024 + wave * 128) * 512 + ((j >> 1) & 1) * 256 + (role == 0 ? 2 * tile : 2 * role - 1) * 64);
#pragma unroll
      for (int t = 0; t < 8; ++t) dma1((lane >> 2) * 512 + (lane & 3) * 16, src + t * 8192, lds0 + L_B + tile * 65536 + wave * 8192 + t * 1024);
    }
    loadx32(A, lane, abase);
    vmwait<32>();
#pragma unroll 1
    for (int tile = 0; tile < ntile; ++tile) {
      const u32 lb = lds0 + L_B + tile * 65536 + wave * 8192;
      const u32 ta = lb + ((lane >> 5) * 8 + ((lane & 15) >> 2)) * 64 + ((lane >> 4) & 1) * 32 + (lane & 3) * 8;
      u32x2 e[16];
      asm volatile("ds_read_b64_tr_b16 %0, %16\n\tds_read_b64_tr_b16 %1, %16 offset:256\n\t"
                   "ds_read_b64_tr_b16 %2, %16 offset:1024\n\tds_read_b64_tr_b16 %3, %16 offset:1280\n\t"
                   "ds_read_b64_tr_b16 %4, %16 offset:2048\n\tds_read_b64_tr_b16 %5, %16 offset:2304\n\t"
                   "ds_read_b64_tr_b16 %6, %16 offset:3072\n\tds_read_b64_tr_b16 %7, %16 offset:3328\n\t"
                   "ds_read_b64_tr_b16 %8, %16 offset:4096\n\tds_read_b64_tr_b16 %9, %16 offset:4352\n\t"
                   "ds_read_b64_tr_b16 %10, %16 offset:5120\n\tds_read_b64_tr_b16 %11, %16 offset:5376\n\t"
                   "ds_read_b64_tr_b16 %12, %16 offset:6144\n\tds_read_b64_tr_b16 %13, %16 offset:6400\n\t"
                   "ds_read_b64_tr_b16 %14, %16 offset:7168\n\tds_read_b64_tr_b16 %15, %16 offset:7424\n\t"
                   "s_waitcnt lgkmcnt(0)"
                   : "=&v"(e[0]), "=&v"(e[1]), "=&v"(e[2]), "=&v"(e[3]), "=&v"(e[4]), "=&v"(e[5]), "=&v"(e[6]), "=&v"(e[7]),
                     "=&v"(e[8]), "=&v"(e[9]), "=&v"(e[10]), "=&v"(e[11]), "=&v"(e[12]), "=&v"(e[13]), "=&v"(e[14]), "=&v"(e[15])
                   : "v"(ta) : "memory");
#pragma unroll
      for (int f = 0; f < 8; ++f)
        *(u32x4*)(smem + L_B + tile * 65536 + (wave * 8 + f) * 1024 + lane * 16) = u32x4{e[2 * f][0], e[2 * f][1], e[2 * f + 1][0], e[2 * f + 1][1]};
      PUBLISH(tile, 257u);
    }
  } else {
#pragma unroll
    for (int q = 0; q < 8; ++q) {
      XWAIT(xflag_of(q));
      load4(A + q * 4, voff, abase + q * 4 * KiB);
    }
  }
  u32x4 yv = {}, y0v = {};
#define SEND_Y(g, t, yv) do { const int tl_ = (ysrc * 4 + (g)) * 8 + yi; \
      st16_wt(own + YS_OFF + (size_t)(tl_ * 8 + m * 2 + kh) * KiB + lane * 16, yv); \
      u32 n_ = 1; if (lane == 0) asm volatile("ds_add_rtn_u32 %0, %1, %0\n\ts_waitcnt lgkmcnt(0)" : "+v"(n_) : "v"(C_BD + 8 + (t) * 4) : "memory"); \
      if (__builtin_amdgcn_readfirstlane(n_) == 7 && lane < 32) \
        st16_wt(own + YE_OFF + tl_ * 512 + lane * 16, *(const u32x4*)(smem + L_EX + (t) * 512 + lane * 16)); } while (0)
  int gam = 0;
  u64 tn = 0;
  bool pend = false;
  const u32 tpos = i - (i > j);
  u64 tg = 0, tg0 = 0;
#define TRY_SEND() do { if (pend && now() >= tg0) { SEND_Y(0, 0, y0v); pend = false; } } while (0)
  const char* const wfl = my + WF_OFF + ((j * 8 + (lane & 1)) * 8 + wave) * 128;
  const char* const wloc = my + slot + WS_OFF + (size_t)(j * 256 + wave * 8) * KiB;
#define WAIT_W(g, mask) do { while (true) { u32 v_ = call; if (lane < 2) v_ = ld_flag(wfl + (g) * 2048); \
      if ((~(u32)__ballot(v_ >= call) & (mask)) == 0) break; \
      if ((mask) == 1u) TRY_SEND(); \
      if (expired<0>(my, t0, it, call)) { lds_st(C_FAIL, 1); break; } } } while (0)
#define DMA_W(g, c, t) dma4(voff, uni(wloc + (g) * 64 * KiB + (c) * 4 * KiB), lds0 + L_B + (t) * 65536 + wave * 8192 + (c) * 4096)
#define GATE(tg_) do { if (tn < (tg_)) while (now() < (tg_)) __builtin_amdgcn_s_sleep(1); } while (0)
#pragma unroll 1
  for (int tile = 0; tile < ntile; ++tile) {
    const int C_T = C_BD + tile * 4;
    gam = role == 0 ? 2 * tile : 2 * role - 1;
    TRY_SEND();
    if (diag) {
      vmwait<0>();
    } else {
      WAIT_W(gam, 1u);
      DMA_W(gam, 0, tile);
      vmwait<0>();
      PUBLISH(tile, 1u);
      WAIT_W(gam, 2u);
      DMA_W(gam, 1, tile);
    }
#pragma unroll
    for (int q = 0; q < 8; ++q)
      asm volatile("" : "+v"(A[q * 4 + 0]), "+v"(A[q * 4 + 1]), "+v"(A[q * 4 + 2]), "+v"(A[q * 4 + 3]) :: "memory");
    f32x16 acc = {};
    const char* bl = smem + L_B + tile * 65536 + kh * 32 * 1024 + lane * 16;
    u32 st = 0;
#define BWAIT(mask) do { while ((st & ((mask) << (kh * 4))) != ((mask) << (kh * 4))) st = lds_ld(C_T); } while (0)
#pragma unroll
    for (int c = 0; c < 2; ++c) {
      if (c == 1) {
        if (!diag) { vmwait<0>(); PUBLISH(tile, 256u); }
        TRY_SEND();
      }
      BWAIT(15u << (c * 8));
      bf16x8 wf[8];
#pragma unroll
      for (int s = 0; s < 8; ++s) wf[s] = *(const bf16x8*)(bl + ((s >> 2) * 8 + c * 4 + (s & 3)) * 1024);
#pragma unroll
      for (int s = 0; s < 16; ++s) {
        __builtin_amdgcn_sched_barrier(0);
        if (s == 8 && c == 1) TRY_SEND();
        acc = __builtin_amdgcn_mfma_f32_32x32x16_bf16(wf[s & 7], __builtin_bit_cast(bf16x8, A[(s >> 2) * 8 + c * 4 + (s & 3)]), acc, 0, 0, 0);
        if (s < 8) wf[s] = *(const bf16x8*)(bl + (16 + (s >> 2) * 8 + c * 4 + (s & 3)) * 1024);
      }
      __builtin_amdgcn_sched_barrier(0);
    }
    TRY_SEND();
    f32x4 o0, o1, k0, k1;
    if (kh == 0) { k0 = acc.s0123; k1 = acc.s4567; o0 = acc.s89ab; o1 = acc.scdef; }
    else         { o0 = acc.s0123; o1 = acc.s4567; k0 = acc.s89ab; k1 = acc.scdef; }
    char* red = smem + (tile == 0 ? L_RED : L_B);
    *(f32x4*)(red + wave * 2048 + lane * 16) = o0;
    *(f32x4*)(red + wave * 2048 + 1024 + lane * 16) = o1;
    tn = now();
    BARRIER();
    const u32 failed = *(const u32*)(smem + L_CTL);
    k0 += *(const f32x4*)(red + (wave ^ 4) * 2048 + lane * 16);
    k1 += *(const f32x4*)(red + (wave ^ 4) * 2048 + 1024 + lane * 16);
    u32 d0 = pack2(k0[0], k0[1]), d1 = pack2(k0[2], k0[3]), d2 = pack2(k1[0], k1[1]), d3 = pack2(k1[2], k1[3]);
    auto s0 = __builtin_amdgcn_permlane32_swap(d0, d2, false, false);
    auto s1 = __builtin_amdgcn_permlane32_swap(d1, d3, false, false);
    yv = u32x4{s0[0], s1[0], s0[1], s1[1]};
    if (failed) yv = u32x4{0x7fc07fc0u, 0x7fc07fc0u, 0x7fc07fc0u, 0x7fc07fc0u};
    if ((lane & 3) == 3) {
      *(u32*)(smem + L_EX + tile * 512 + (m * 2 + kh) * 64 + (lane >> 2) * 4) = (yv[3] >> 16) | tag;
      yv[3] = (yv[3] & 0xffffu) | tag;
    }
    tg = t0 + G0 + ((gam < 2 ? gam ^ 1 : gam) * 7 + tpos) * GSTEP / 10;
    if (tg < tn + tpos * GSTEP / 10) tg = tn + tpos * GSTEP / 10;
    if (diag) tg = 0;
    if (tile + 1 < ntile) { pend = true; y0v = yv; tg0 = tg; }
  }
  if (pend) { GATE(tg0); SEND_Y(0, 0, y0v); }
  GATE(tg);
  SEND_Y(gam, ntile - 1, yv);
  if (tid == 0) __hip_atomic_store((gu32*)cntp, call, RLX_AGT);
}

extern "C" void kernel_launch(int rank, int nranks, ncclComm_t comm, hipStream_t stream,
                              void* const* d_in, const int* in_sizes, int n_in,
                              void* d_out, int out_size, const int* in_dtypes, int out_dtype,
                              void* const* d_out_all, void* const* d_ws_all, size_t ws_bytes) {
  if (nranks != NR || n_in < 2 || ws_bytes < WS_NEED || (unsigned)rank >= (unsigned)NR ||
      in_sizes[0] != 256 * 2048 || in_sizes[1] != 2048 * 256 || out_size != 2048 * 256) {
    fprintf(stderr, "kernel_launch: unexpected problem shape\n");
    return;
  }
  static bool attr_done = false;
  if (!attr_done) {
    HIP_CHECK(hipFuncSetAttribute((const void*)ag_gemm_kernel, hipFuncAttributeMaxDynamicSharedMemorySize, L_BYTES));
    attr_done = true;
  }
  Params p;
  p.x = (const uint16_t*)d_in[0];
  p.w = (const uint16_t*)d_in[1];
  p.out = (uint16_t*)d_out;
  p.my = (char*)d_ws_all[rank];
  const unsigned grp = 0x4Du, same = (grp >> rank) & 1 ? grp : ~grp & 0xffu;
  p.dest = 0;
  for (int q = 0, lo = 0, hi = 4; q < NR; ++q) {
    const int idx = (same >> q) & 1 ? lo++ : hi++;
    p.ws[idx] = (char*)d_ws_all[q];
    p.dest |= (u32)q << (4 * idx);
  }
  p.rank = rank;
  ag_gemm_kernel<<<256, 512, L_BYTES, stream>>>(p);
  HIP_CHECK(hipGetLastError());
}
```
